```python
import numpy as np

kernel_name = "gpu_gemm_a2a_m4096_k4096_n4096_bf16_none"


ABSMAX_RTOL = 0.008

COMPUTE_DTYPE = "bfloat16"
EPILOGUE = "none"
IN_QUANT = None
OUT_QUANT = None
IN_DTYPES = {"x": "bfloat16", "w_mat": "bfloat16"}
WORLD_SIZE = 8

def _to_bf16(x):
    u = np.ascontiguousarray(x, dtype=np.float32).view(np.uint32)
    return ((u + 0x7FFF + ((u >> 16) & 1)) & 0xFFFF0000).view(np.float32)

def setup_inputs():

    x = _to_bf16(np.random.randn(4096, 4096).astype(np.float32))
    w_mat = _to_bf16((np.random.randn(4096, 4096) / np.sqrt(4096)).astype(np.float32))
    return {"x": x, "w_mat": w_mat}

def _epilogue(y):
    return y

def reference(x, w_mat):
    y = (x @ w_mat).astype(np.float32)
    return _epilogue(y)
```

```cpp
#include <hip/hip_runtime.h>
#include <rccl/rccl.h>
#include <cstdio>
#include <cstdint>

#define HIP_CHECK(expr) do { hipError_t _e = (expr); if (_e != hipSuccess) { \
    fprintf(stderr, "HIP error %s at %s:%d\n", hipGetErrorString(_e), __FILE__, __LINE__); \
    return; } } while (0)

namespace {
typedef short s16x4 __attribute__((ext_vector_type(4)));
typedef short s16x8 __attribute__((ext_vector_type(8)));
typedef __bf16 bf16x8 __attribute__((ext_vector_type(8)));
typedef __bf16 bf16x2 __attribute__((ext_vector_type(2)));
typedef float f32x16 __attribute__((ext_vector_type(16)));
typedef float f32x2 __attribute__((ext_vector_type(2)));
typedef unsigned u32x4 __attribute__((ext_vector_type(4)));
typedef unsigned u32x2 __attribute__((ext_vector_type(2)));
typedef int i32x4 __attribute__((ext_vector_type(4)));
#define LAS __attribute__((address_space(3)))

constexpr int NR = 8;
constexpr int MS = 512;
constexpr int KD = 4096, ND = 4096;
constexpr int NS = 512;
constexpr int BM = 128, BN = 64, BK = 64;
constexpr int NST = 6;
constexpr int XB = BM * BK * 2, WB = BK * BN * 2, STB = XB + WB;
constexpr int NWG = (MS / BM) * (ND / BN);
constexpr int NKT = KD / BK;
constexpr unsigned long long DEADLINE = 40000000ull;

constexpr size_t WS_EPOCH = 0;
constexpr size_t WS_TMO = 4096;
constexpr size_t WS_FLAGS = 8192;

struct Params {
  const void* x; const void* w;
  void* out[NR]; void* ws[NR];
  int rank;
};

__device__ __forceinline__ void dma16(unsigned lds, unsigned voff, i32x4 srd, unsigned soff) {
  asm volatile("s_mov_b32 m0, %0\n\ts_nop 0\n\tbuffer_load_dwordx4 %1, %2, %3 offen lds"
               :: "s"(lds), "v"(voff), "s"(srd), "s"(soff) : "memory");
}
__device__ __forceinline__ i32x4 make_srd(const void* p, unsigned bytes) {
  uint64_t a = (uint64_t)p;
  i32x4 r; r[0] = (int)(unsigned)a; r[1] = (int)(unsigned)((a >> 32) & 0xffff); r[2] = (int)bytes; r[3] = 0x00020000;
  return r;
}
__device__ __forceinline__ unsigned pack_bf16(float a, float b) {
  f32x2 v = {a, b};
  bf16x2 r = __builtin_convertvector(v, bf16x2);
  return __builtin_bit_cast(unsigned, r);
}

__global__ __launch_bounds__(256, 1) void gemm_a2a(Params p) {
  __shared__ __attribute__((aligned(1024))) char smem[NST * STB];
  const int tid = threadIdx.x, lane = tid & 63;
  const int wid = __builtin_amdgcn_readfirstlane(tid >> 6);
  const int bid = blockIdx.x;
  const int g = bid & 7, idx = bid >> 3;
  const int dst = (g + p.rank) & 7;
  const int mt = idx & 3, nt = idx >> 2;
  const int m0 = mt * BM, n0 = dst * NS + nt * BN;

  unsigned* wsl = (unsigned*)p.ws[p.rank];
  const unsigned call = wsl[WS_EPOCH / 4 + bid] + 1;

  const i32x4 xsrd = make_srd(p.x, MS * KD * 2), wsrd = make_srd(p.w, KD * ND * 2);
  const unsigned lbase = (unsigned)(uintptr_t)smem;
  unsigned xoff[4], woff[2];
#pragma unroll
  for (int i = 0; i < 4; ++i) {
    int r = (wid * 4 + i) * 8 + (lane >> 3), c = (lane & 7) ^ ((r >> 1) & 7);
    xoff[i] = (unsigned)((m0 + r) * KD * 2 + c * 16);
  }
#pragma unroll
  for (int i = 0; i < 2; ++i) {
    int r = (wid * 2 + i) * 8 + (lane >> 3), c = (lane & 7) ^ ((((r >> 1) & 1) << 2) | (((r >> 3) & 1) << 1));
    woff[i] = (unsigned)(r * ND * 2 + n0 * 2 + c * 16);
  }
  auto stage = [&](int kt, int slot) {
    unsigned lb = lbase + slot * STB;
#pragma unroll
    for (int i = 0; i < 4; ++i) dma16(lb + (wid * 4 + i) * 1024, xoff[i], xsrd, kt * BK * 2);
#pragma unroll
    for (int i = 0; i < 2; ++i) dma16(lb + XB + (wid * 2 + i) * 1024, woff[i], wsrd, kt * BK * ND * 2);
  };
  const int wm = wid & 1, wn = wid >> 1;
  const int l31 = lane & 31, h = lane >> 5;
  unsigned xr[2][4];
#pragma unroll
  for (int i = 0; i < 2; ++i)
#pragma unroll
    for (int ks = 0; ks < 4; ++ks) {
      int r = wm * 64 + i * 32 + l31, c = (2 * ks + h) ^ ((r >> 1) & 7);
      xr[i][ks] = r * 128 + c * 16;
    }
  unsigned wr[4][2];
  {
    int gq = lane >> 4, hh = gq >> 1, cb = gq & 1, q = (lane & 15) >> 2, pp = lane & 3;
#pragma unroll
    for (int ks = 0; ks < 4; ++ks)
#pragma unroll
      for (int t = 0; t < 2; ++t) {
        int r = 16 * ks + 8 * hh + 4 * t + q;
        int ch = wn * 4 + cb * 2 + (pp >> 1);
        int c = ch ^ ((((r >> 1) & 1) << 2) | (((r >> 3) & 1) << 1));
        wr[ks][t] = XB + r * 128 + c * 16 + (pp & 1) * 8;
      }
  }
  f32x16 acc[2] = {};
#pragma unroll
  for (int s = 0; s < NST - 1; ++s) stage(s, s);
  int slot = 0;
  for (int kt = 0; kt < NKT; ++kt) {
    if (kt + NST - 1 <= NKT - 0 - 0 && kt + NST - 2 < NKT) asm volatile("s_waitcnt vmcnt(%0)" :: "n"(6 * (NST - 2)) : "memory");
    else asm volatile("s_waitcnt vmcnt(0)" ::: "memory");
    asm volatile("s_waitcnt lgkmcnt(0)" ::: "memory");
    __builtin_amdgcn_s_barrier();
    asm volatile("" ::: "memory");
    int pslot = slot == 0 ? NST - 1 : slot - 1;
    if (kt + NST - 1 < NKT) stage(kt + NST - 1, pslot);
    const char* sb = smem + slot * STB;
    s16x8 xf[2][4]; s16x4 wf[4][2];
#pragma unroll
    for (int ks = 0; ks < 4; ++ks) {
      wf[ks][0] = __builtin_amdgcn_ds_read_tr16_b64_v4i16((LAS s16x4*)(sb + wr[ks][0]));
      wf[ks][1] = __builtin_amdgcn_ds_read_tr16_b64_v4i16((LAS s16x4*)(sb + wr[ks][1]));
      xf[0][ks] = *(const s16x8*)(sb + xr[0][ks]);
      xf[1][ks] = *(const s16x8*)(sb + xr[1][ks]);
    }
#pragma unroll
    for (int ks = 0; ks < 4; ++ks) {
      s16x8 w8 = __builtin_shufflevector(wf[ks][0], wf[ks][1], 0, 1, 2, 3, 4, 5, 6, 7);
#pragma unroll
      for (int i = 0; i < 2; ++i)
        acc[i] = __builtin_amdgcn_mfma_f32_32x32x16_bf16(__builtin_bit_cast(bf16x8, w8), __builtin_bit_cast(bf16x8, xf[i][ks]), acc[i], 0, 0, 0);
    }
    slot = slot == NST - 1 ? 0 : slot + 1;
  }
  asm volatile("s_waitcnt lgkmcnt(0)" ::: "memory");
  __builtin_amdgcn_s_barrier();
  asm volatile("" ::: "memory");
#pragma unroll
  for (int i = 0; i < 2; ++i) {
    int m = wm * 64 + i * 32 + l31;
#pragma unroll
    for (int q = 0; q < 4; ++q) {
      u32x2 v; v[0] = pack_bf16(acc[i][4 * q], acc[i][4 * q + 1]); v[1] = pack_bf16(acc[i][4 * q + 2], acc[i][4 * q + 3]);
      int ch = wn * 4 + q;
      *(u32x2*)(smem + m * 128 + ((ch ^ (m & 7)) * 16) + h * 8) = v;
    }
  }
  __syncthreads();
  {
    char* ob = (char*)p.out[dst] + (size_t)(p.rank * MS + m0) * (NS * 2) + nt * BN * 2;
#pragma unroll
    for (int i = 0; i < 4; ++i) {
      int m = i * 32 + (tid >> 3), ch = tid & 7;
      u32x4 v = *(const u32x4*)(smem + m * 128 + ((ch ^ (m & 7)) * 16));
      char* dp = ob + (size_t)m * (NS * 2) + ch * 16;
      asm volatile("global_store_dwordx4 %0, %1, off sc0 sc1\n\ts_nop 1" :: "v"(dp), "v"(v) : "memory");
    }
  }
  asm volatile("s_waitcnt vmcnt(0)" ::: "memory");
  __syncthreads();
  if (tid == 0) {
    wsl[WS_EPOCH / 4 + bid] = call;
    if (dst != p.rank) {
      __builtin_amdgcn_fence(__ATOMIC_RELEASE, "");
      asm volatile("s_waitcnt vmcnt(0)" ::: "memory");
      unsigned* fl = (unsigned*)((char*)p.ws[dst] + WS_FLAGS) + p.rank * 32 + idx;
      __hip_atomic_store(fl, call, __ATOMIC_RELAXED, __HIP_MEMORY_SCOPE_SYSTEM);
    }
  }
  if (bid == 0) {
    unsigned* fl = (unsigned*)((char*)wsl + WS_FLAGS) + tid;
    unsigned* tmo = (unsigned*)((char*)wsl + WS_TMO);
    bool mine = (tid >> 5) != p.rank;
    unsigned long long t0 = __builtin_amdgcn_s_memrealtime();
    bool timed_out = false;
    while (true) {
      unsigned v = mine ? __hip_atomic_load(fl, __ATOMIC_RELAXED, __HIP_MEMORY_SCOPE_SYSTEM) : call;
      if (__all((int)(v - call) >= 0)) break;
      if (__builtin_amdgcn_s_memrealtime() - t0 > DEADLINE) { timed_out = true; break; }
      __builtin_amdgcn_s_sleep(1);
    }
    if (timed_out) {
      __hip_atomic_store(tmo, call, __ATOMIC_RELAXED, __HIP_MEMORY_SCOPE_AGENT);
      if (lane == 0) ((unsigned short*)p.out[p.rank])[(size_t)(p.rank * MS) * NS + wid] = 0x7fc0;
    }
    __builtin_amdgcn_fence(__ATOMIC_ACQUIRE, "");
  }
}
}

extern "C" void kernel_launch(int rank, int nranks, ncclComm_t comm, hipStream_t stream,
                              void* const* d_in, const int* in_sizes, int n_in,
                              void* d_out, int out_size, const int* in_dtypes, int out_dtype,
                              void* const* d_out_all, void* const* d_ws_all, size_t ws_bytes) {
  if (nranks != NR || (unsigned)rank >= (unsigned)NR || ws_bytes < 65536) { fprintf(stderr, "unsupported config\n"); return; }
  Params p;
  p.x = d_in[0]; p.w = d_in[1]; p.rank = rank;
  for (int i = 0; i < NR; ++i) { p.out[i] = d_out_all[i]; p.ws[i] = d_ws_all[i]; }
  gemm_a2a<<<NWG, 256, 0, stream>>>(p);
  HIP_CHECK(hipGetLastError());
}
```

```cpp
#include <hip/hip_runtime.h>
#include <rccl/rccl.h>
#include <cstdio>
#include <cstdint>

#define HIP_CHECK(expr) do { hipError_t _e = (expr); if (_e != hipSuccess) { \
    fprintf(stderr, "HIP error %s at %s:%d\n", hipGetErrorString(_e), __FILE__, __LINE__); \
    return; } } while (0)

namespace {
typedef short s16x4 __attribute__((ext_vector_type(4)));
typedef __bf16 bf16x8 __attribute__((ext_vector_type(8)));
typedef __bf16 bf16x2 __attribute__((ext_vector_type(2)));
typedef float f32x4 __attribute__((ext_vector_type(4)));
typedef float f32x2 __attribute__((ext_vector_type(2)));
typedef unsigned u32x4 __attribute__((ext_vector_type(4)));
typedef int i32x4 __attribute__((ext_vector_type(4)));
#define LAS __attribute__((address_space(3)))
#define GAS __attribute__((address_space(1)))
#define DI __device__ __forceinline__

constexpr int NR = 8, MS = 512, KD = 4096, ND = 4096, NS = 512;
constexpr int NWG = 256;
constexpr unsigned long long DEADLINE = 40000000ull;

constexpr unsigned WS_EPOCH = 0;
constexpr unsigned WS_TMO = 4096;
constexpr unsigned WS_FLAGS = 8192;
constexpr unsigned WS_PART = 1u << 20;
constexpr unsigned PART_BYTES = 256u * 16u * 4096u;
constexpr size_t WS_NEED = (size_t)WS_PART + PART_BYTES;

struct Params {
  const void* x; const void* w;
  unsigned* wsl;
  int rank;
  void* out[NR]; void* ws[NR];
};

DI i32x4 make_srd(const void* p, unsigned bytes) {
  uint64_t a = (uint64_t)p;
  i32x4 r; r[0] = (int)(unsigned)a; r[1] = (int)(unsigned)((a >> 32) & 0xffff); r[2] = (int)bytes; r[3] = 0x00020000;
  return r;
}
template <int LI> DI void set_m0(unsigned lb) { asm volatile("s_add_i32 m0, %0, %1" :: "s"(lb), "i"(LI) : "scc"); }
DI void dma16(unsigned voff, i32x4 srd, unsigned soff) { asm volatile("buffer_load_dwordx4 %0, %1, %2 offen lds" :: "v"(voff), "s"(srd), "s"(soff) : "memory"); }
DI unsigned pack_bf16(float a, float b) { f32x2 v = {a, b}; return __builtin_bit_cast(unsigned, __builtin_convertvector(v, bf16x2)); }
template <int OFF, int LOFF> DI void st_part(u32x4 v, unsigned voff, i32x4 srd, unsigned soff, unsigned lds, unsigned notown) {
  asm volatile("s_cmp_eq_u32 %6, 0\n\ts_cbranch_scc1 1f\n\tbuffer_store_dwordx4 %0, %1, %2, %3 offen offset:%4 sc1\n\ts_branch 2f\n1:\n\tds_write_b128 %5, %0 offset:%7\n2:\n\ts_nop 0"
               :: "v"(v), "v"(voff), "s"(srd), "s"(soff), "i"(OFF), "v"(lds), "s"(notown), "i"(LOFF) : "memory", "scc");
}
DI void ld_part(u32x4& v, unsigned voff, i32x4 srd, unsigned soff) {
  asm volatile("buffer_load_dwordx4 %0, %1, %2, %3 offen sc1" : "=v"(v) : "v"(voff), "s"(srd), "s"(soff) : "memory");
}
DI void st_out(u32x4 v, unsigned voff, i32x4 srd, unsigned soff) {
  asm volatile("buffer_store_dwordx4 %0, %1, %2, %3 offen sc0 sc1\n\ts_nop 1" :: "v"(v), "v"(voff), "s"(srd), "s"(soff) : "memory");
}
DI bf16x8 cat(s16x4 a, s16x4 b) { return __builtin_bit_cast(bf16x8, __builtin_shufflevector(a, b, 0, 1, 2, 3, 4, 5, 6, 7)); }
DI float asf(unsigned u) { return __builtin_bit_cast(float, u); }

__global__ __launch_bounds__(256, 1) void gemm_a2a(Params p) {
  __shared__ __attribute__((aligned(1024))) char smem[160 * 1024];
  const int tid = threadIdx.x, lane = tid & 63;
  const int wid = __builtin_amdgcn_readfirstlane(tid >> 6);
  const int bid = blockIdx.x;
  unsigned* wsl; const void* px; const void* pw; int prank;
  asm volatile("s_load_dwordx2 %0, %4, %5\n\ts_load_dwordx2 %1, %4, %6\n\ts_load_dwordx2 %2, %4, %7\n\ts_load_dword %3, %4, %8" : "=&s"(px), "=&s"(pw), "=&s"(wsl), "=&s"(prank)
               : "s"(__builtin_amdgcn_kernarg_segment_ptr()), "i"(__builtin_offsetof(Params, x)), "i"(__builtin_offsetof(Params, w)), "i"(__builtin_offsetof(Params, wsl)), "i"(__builtin_offsetof(Params, rank)));
  __builtin_amdgcn_sched_barrier(0);
  unsigned call;
  const int q = bid & 3, idx = bid >> 3;
  const int mb = idx & 3, cgl = idx >> 2, cg = ((bid >> 2) & 1) * 8 + cgl;
  const int dstrel = cg >> 1, nb = cg & 1;
  const int cls = (mb + cgl) & 3;
  unsigned ptab;
#define PERM(s) ((s) == 0 ? (unsigned)cls : ((ptab >> (4 * (s))) & 7u))
#define COFF(c) ((unsigned)((c) * 256 + nb * 128))
  void* outp; void* wsd;
  const unsigned lbase = (unsigned)(uintptr_t)smem;
  const unsigned lbx = lbase + wid * 8192, lbw = lbase + wid * 4096;
  unsigned xoff[8], woff[4][2], ksx[8], ksw[2][8];
#pragma unroll
  for (int i = 0; i < 2; ++i) {
    int r = 8 * i + (lane >> 3);
    xoff[i] = (unsigned)((mb * 128 + wid * 32 + r) * (KD * 2) + q * 2048 + (((lane & 7) ^ ((r >> 1) & 7)) * 16));
    int c = (lane & 7) ^ ((((r >> 1) & 1) << 2) | (i << 1));
    woff[0][i] = (unsigned)((q * 1024 + wid * 32 + r) * (ND * 2) + c * 16) + COFF(0);
  }
#pragma unroll
  for (int i = 2; i < 8; ++i) xoff[i] = xoff[i & 1] + ((i >> 1) & 1) * 16 * KD * 2 + (i >> 2) * 128;
  ksx[0] = PERM(0) << 8; ksw[0][0] = PERM(0) << 20; ksw[1][0] = ksw[0][0] + 16 * ND * 2;
  asm volatile("" : "+v"(xoff[0]), "+v"(xoff[1]), "+v"(xoff[2]), "+v"(xoff[3]), "+v"(xoff[4]), "+v"(xoff[5]), "+v"(xoff[6]), "+v"(xoff[7]), "+v"(woff[0][0]), "+v"(woff[0][1]));
  unsigned toff = (unsigned)(q * 8 + wid) << 20, toff2 = toff + (4u << 20);
  asm volatile("" : "+s"(ksx[0]), "+s"(ksw[0][0]), "+s"(ksw[1][0]), "+s"(toff), "+s"(toff2));
  unsigned tw[3], tch[3];
  asm volatile("s_waitcnt lgkmcnt(0)\n\ts_load_dword %4, %2, %7\n\ts_load_dword %5, %2, %8\n\ts_load_dword %6, %0, %9"
               : "+s"(wsl), "+s"(px), "+s"(pw), "+s"(prank), "=&s"(tw[0]), "=&s"(tw[1]), "=&s"(tw[2]) : "s"(toff), "s"(toff2), "i"(WS_TMO));
  const int rank = prank & 7;
  const int dst = (rank + 1 + dstrel) & 7;
  const i32x4 xsrd = make_srd(px, MS * KD * 2), wsrd = make_srd((const char*)pw + dst * NS * 2, (unsigned)KD * ND * 2u - dst * NS * 2);
#define MX(L) set_m0<(L)>(lbx)
#define MW(L) set_m0<(L)>(lbw)
#define NOP1 asm volatile("s_nop 0")
#define DX(i, s) dma16(xoff[i], xsrd, ksx[s])
#define DW(i, c, s) dma16(woff[c][(i) & 1], wsrd, ksw[(i) >> 1][s])
#define WL asm volatile("s_waitcnt lgkmcnt(0)" ::: "memory")
#define WV(n) asm volatile("s_waitcnt vmcnt(%0)" :: "n"(n) : "memory")
#define BAR do { asm volatile("" ::: "memory"); __builtin_amdgcn_s_barrier(); asm volatile("" ::: "memory"); } while (0)
#define SB __builtin_amdgcn_sched_barrier(0)
#define RW0(r, win, ct, off) wl[r] = __builtin_amdgcn_ds_read_tr16_b64_v4i16((LAS s16x4*)(uintptr_t)(wrd[win][ct] + (off)))
#define RW1(r, win, ct, off) wh[r] = __builtin_amdgcn_ds_read_tr16_b64_v4i16((LAS s16x4*)(uintptr_t)(wrd[win][ct] + (off)))
#define RX(i, k, off) xf[i] = *(LAS bf16x8*)(uintptr_t)(xrd[k] + (off))
#define TDELAY __builtin_amdgcn_s_sleep(6)
#define MF(a, rt, ct, r, xi) asm volatile("v_mfma_f32_16x16x32_bf16 %0, %1, %2, %0" : "+v"(acc[a][rt][ct]) : "v"(cat(wl[r], wh[r])), "a"(xf[xi]))
#define MZ(a, rt, ct, r, xi) asm volatile("v_mfma_f32_16x16x32_bf16 %0, %1, %2, 0" : "=&v"(acc[a][rt][ct]) : "v"(cat(wl[r], wh[r])), "a"(xf[xi]))
#define MFW(a, rt, ct, r, xi) asm volatile("v_mfma_f32_16x16x32_bf16 %0, %1, %2, %0" : "+v"(acc[a][rt][ct]) : "v"(cat(wl[r], wh[r])), "a"(xf[xi]), "v"(wh[(r) + 1]))
#define ACC(c, rt, ct) acc[c][rt][ct]
#define PK0(c, rt, P) pu = (__builtin_bit_cast(unsigned, ACC(c, rt, 2 * (P))[0]) & himask) | tagl
#define PK1(c, rt, P) pk[1] = pack_bf16(ACC(c, rt, 2 * (P))[2], ACC(c, rt, 2 * (P))[3])
#define PK2(c, rt, P) pk[0] = pack_bf16(asf(pu), ACC(c, rt, 2 * (P))[1])
#define PK3(c, rt, P) pk[2] = pack_bf16(ACC(c, rt, 2 * (P) + 1)[0], ACC(c, rt, 2 * (P) + 1)[1])
#define PK4(c, rt, P) pk[0] = (pk[0] & clrm) | tagp
#define PK5(c, rt, P) pk[3] = pack_bf16(ACC(c, rt, 2 * (P) + 1)[2], ACC(c, rt, 2 * (P) + 1)[3])
#define PK6(c, rt, P) st_part<(rt) * 2048 + (P) * 1024, (c) * 4096 + (rt) * 2048 + (P) * 1024>(pk, pst_v, psrd, tso[c], poa, own)
#define GL(c, s) ld_part(gv[s], pld_v[c], psrd, gso[s])
#define GO(c) gown[(c) & 1] = *(LAS u32x4*)(uintptr_t)(goa + (c) * 4096)
#define GW(n) asm volatile("s_waitcnt vmcnt(%3)" : "+v"(gv[0]), "+v"(gv[1]), "+v"(gv[2]) : "n"(n) : "memory")
#define GK0 chk = (gv[0][0] ^ gv[1][0]) | (gv[0][0] ^ gv[2][0])
#define GK1 chk = chk | (gv[0][0] ^ tag)
#define STALE() (__builtin_amdgcn_ballot_w64((chk & 1u) != 0) != 0)
#define GIVE_UP(t0_) (__builtin_amdgcn_s_memrealtime() - (t0_) > DEADLINE || __builtin_amdgcn_readfirstlane(__hip_atomic_load(tmo, __ATOMIC_RELAXED, __HIP_MEMORY_SCOPE_AGENT)) == call)
#define GC(c) do { if (__builtin_expect(STALE(), 0)) { \
    const unsigned long long t0_ = __builtin_amdgcn_s_memrealtime(); \
    for (unsigned n_ = 1;; ++n_) { GL(c, 0); GL(c, 1); GL(c, 2); GW(0); GK0; GK1; if (!STALE()) break; \
      if ((n_ & 15u) == 0 && GIVE_UP(t0_)) { \
        __hip_atomic_store(tmo, call, __ATOMIC_RELAXED, __HIP_MEMORY_SCOPE_AGENT); \
        gv[0] = u32x4{0x7fc07fc0u, 0x7fc07fc0u, 0x7fc07fc0u, 0x7fc07fc0u}; break; } } \
    __builtin_amdgcn_s_waitcnt(0); asm volatile("s_nop 3" : "+v"(gv[0]), "+v"(gv[1]), "+v"(gv[2])); } } while (0)
#define GM(s, par) do { if ((s) == 0) { asm volatile("v_mfma_f32_16x16x32_bf16 %0, %1, %2, 0" : "=&v"(gd[0]) : "v"(sel[0]), "v"(gown[par])); asm volatile("v_mfma_f32_16x16x32_bf16 %0, %1, %2, 0" : "=&v"(gd[1]) : "v"(sel[1]), "v"(gown[par])); } \
    else { asm volatile("v_mfma_f32_16x16x32_bf16 %0, %1, %2, %0" : "+v"(gd[0]) : "v"(sel[0]), "v"(gv[(s) > 0 ? (s) - 1 : 0])); asm volatile("v_mfma_f32_16x16x32_bf16 %0, %1, %2, %0" : "+v"(gd[1]) : "v"(sel[1]), "v"(gv[(s) > 0 ? (s) - 1 : 0])); } } while (0)
#define NOP8 asm volatile("s_nop 7" : "+v"(gd[0]), "+v"(gd[1]))
#define NOPACC do { asm volatile("s_nop 7" ::: "memory"); SB; } while (0)
#define GP(i) ov[i] = pack_bf16(gd[(i) >> 1][2 * ((i) & 1)], gd[(i) >> 1][2 * ((i) & 1) + 1])
#define GT(c) st_out(ov, out_v, osrd, oso[c])

  bf16x8 xf[64];
  f32x4 acc[4][2][4];
  const unsigned own = (unsigned)(wid ^ q);
  s16x4 wl[8], wh[8];
  unsigned pu, chk = 0, himask, tagp, clrm;
  u32x4 pk, gv[3], gown[2], ov, sel[2];
  f32x4 gd[2];
  unsigned wrd[2][4], xrd[2], poa, goa;
  unsigned tag, tagl, tso[4], oso[4], pst_v, gso[4], pld_v[4], out_v; GAS unsigned* tmo; GAS unsigned* flp;
  i32x4 psrd, osrd;
  const int g4 = lane >> 4, m16 = lane & 15, qq = m16 >> 2, pp = lane & 3;
#define SETUP1 do { \
    asm volatile("s_load_dword %0, %3, %4\n\ts_load_dwordx2 %1, %5, %6 offset:%7\n\ts_load_dwordx2 %2, %5, %6 offset:%8" : "=&s"(call), "=&s"(outp), "=&s"(wsd) \
                 : "s"(wsl), "s"(bid * 4), "s"(__builtin_amdgcn_kernarg_segment_ptr()), "s"(dst * 8), "i"(__builtin_offsetof(Params, out)), "i"(__builtin_offsetof(Params, ws)));   \
    { const unsigned lo_ = (cls & 1) ? 0x76403521u : 0x76532140u, hi_ = (cls & 1) ? 0x65472103u : 0x75416032u; \
      ptab = (cls & 2) ? hi_ : lo_; asm volatile("" : "+s"(ptab)); } \
    { int r = 8 * g4 + qq, sw = (((r >> 1) & 1) << 2) | ((g4 & 1) << 1); \
      _Pragma("unroll") for (int c = 0; c < 4; ++c) { wrd[0][c] = lbase + 32768 + r * 128 + (((2 * c + (pp >> 1)) ^ sw) * 16) + (pp & 1) * 8; wrd[1][c] = wrd[0][c] + 65536; asm volatile("" : "+v"(wrd[0][c]), "+v"(wrd[1][c])); } } \
    _Pragma("unroll") for (int k = 0; k < 2; ++k) { xrd[k] = lbx + m16 * 128 + (((4 * k + g4) ^ (m16 >> 1)) * 16); asm volatile("" : "+v"(xrd[k])); } \
    _Pragma("unroll") for (int s_ = 1; s_ < 8; ++s_) { ksx[s_] = PERM(s_) << 8; ksw[0][s_] = PERM(s_) << 20; ksw[1][s_] = ksw[0][s_] + 16 * ND * 2; asm volatile("" : "+s"(ksx[s_]), "+s"(ksw[0][s_]), "+s"(ksw[1][s_])); } \
  } while (0)
#define SETUP2 do { \
    poa = lbase + lane * 16; goa = lbase + (unsigned)((wid >> 1) * 2048 + ((lane >> 3) & 1) * 1024 + g4 * 256 + ((wid & 1) * 8 + (lane & 7)) * 16); \
    _Pragma("unroll") for (int hh = 0; hh < 2; ++hh) { const int e = 4 * (qq >> 1) + pp; const unsigned one = (g4 == 2 * (qq & 1) + hh) ? (0x3f80u << (16 * (e & 1))) : 0u; \
      _Pragma("unroll") for (int d = 0; d < 4; ++d) sel[hh][d] = (e >> 1) == d ? one : 0u; asm volatile("" : "+v"(sel[hh])); } \
    _Pragma("unroll") for (int i_ = 0; i_ < 2; ++i_) { woff[1][i_] = woff[0][i_] + 256; woff[2][i_] = woff[0][i_] + 512; woff[3][i_] = woff[0][i_] + 768; asm volatile("" : "+v"(woff[1][i_]), "+v"(woff[2][i_]), "+v"(woff[3][i_])); } \
    out_v = (unsigned)((rank * MS + mb * 128 + q * 32 + wid * 8 + (lane & 7)) * (NS * 2) + ((lane >> 3) & 1) * 64 + g4 * 16); \
    asm volatile("" : "+v"(out_v), "+v"(poa), "+v"(goa)); \
  } while (0)
#define SETUP3 do { \
    const unsigned gl_ = goa - lbase; \
    asm volatile("s_waitcnt lgkmcnt(0)" : "+s"(call), "+s"(outp), "+s"(wsd) : "s"(tw[0]), "s"(tw[1]), "s"(tw[2])); call += 1; \
    tag = call & 1u; tagl = (tag || !own) ? 0u : 0xffffu; tagp = own ? tag : 0u; clrm = own ? 0xfffffffeu : 0xffffffffu; himask = own ? 0xffff0000u : 0xffffffffu; \
    psrd = make_srd((char*)wsl + WS_PART, PART_BYTES); osrd = make_srd(outp, NR * MS * NS * 2); \
    tmo = (GAS unsigned*)wsl + WS_TMO / 4; \
    flp = (GAS unsigned*)wsd + WS_FLAGS / 4 + ((q * 4 + mb) * 2 + nb) * 32 + (dstrel != 7 ? rank * 4 + wid : (lane & 31)); asm volatile("" : "+v"(flp)); \
    const unsigned tile_so = (unsigned)(mb * 16 + cg) << 18; \
    const unsigned osh = tag ? 14u : 12u, ssh = tag ? 12u : 14u; \
    pst_v = ((unsigned)wid << osh) + ((unsigned)q << ssh) + lane * 16; \
    _Pragma("unroll") for (int c_ = 0; c_ < 4; ++c_) { gso[c_] = tile_so + ((unsigned)((q + 1 + c_) & 3) << ssh); tso[c_] = tile_so + c_ * 65536; oso[c_] = COFF(c_); \
      pld_v[c_] = ((unsigned)q << osh) + c_ * 65536 + gl_; \
      asm volatile("" : "+s"(gso[c_]), "+s"(tso[c_]), "+s"(oso[c_]), "+v"(pld_v[c_])); } \
    asm volatile("" : "+v"(pst_v), "+s"(tag), "+v"(tagl), "+v"(tagp), "+v"(clrm), "+v"(himask), "+s"(psrd), "+s"(osrd)); \
    asm volatile("s_load_dword %0, %3, %4\n\ts_load_dword %1, %5, %6\n\ts_load_dword %2, %7, %8" : "=&s"(tch[0]), "=&s"(tch[1]), "=&s"(tch[2]) \
                 : "s"(outp), "s"((unsigned)((rank * MS + mb * 128 + q * 32 + wid * 8) * (NS * 2))), "s"(wsd), "s"(WS_FLAGS + 4096u), "s"(wsl), "s"(WS_PART + tile_so + wid * 65536u)); \
  } while (0)
#define TOUCHW asm volatile("s_waitcnt lgkmcnt(0)" :: "s"(tch[0]), "s"(tch[1]), "s"(tch[2]))

SB;
MX(0); NOP1; DX(0,0);
MX(1024); NOP1; DX(1,0);
MX(2048); NOP1; DX(2,0);
MX(3072); NOP1; DX(3,0);
MX(4096); NOP1; DX(4,0);
MX(5120); NOP1; DX(5,0);
MX(6144); NOP1; DX(6,0);
MX(7168); NOP1; DX(7,0);
MW(98304); NOP1; DW(0,0,0);
MW(99328); NOP1; DW(1,0,0);
MW(100352); NOP1; DW(2,0,0);
MW(101376); NOP1; DW(3,0,0);
SB; SETUP1; SB; SETUP2; SB;
MW(114688); NOP1; DW(0,0,1);
MW(115712); NOP1; DW(1,0,1);
MW(116736); NOP1; DW(2,0,1);
MW(117760); NOP1; DW(3,0,1);
MX(32768); NOP1; DX(0,1);
MX(33792); NOP1; DX(1,1);
MX(34816); NOP1; DX(2,1);
MX(35840); NOP1; DX(3,1);
MX(36864); NOP1; DX(4,1);
MX(37888); NOP1; DX(5,1);
MX(38912); NOP1; DX(6,1);
MX(39936); NOP1; DX(7,1);
SB; SETUP3; SB;
WV(12); BAR;
RW0(0,1,0,0);
RW1(0,1,0,512);
RX(0,0,0);
RW0(1,1,1,0);
RW1(1,1,1,512);
RX(1,0,2048);
RW0(2,1,2,0);
RW1(2,1,2,512);
RX(2,1,0);
RW0(3,1,3,0);
RW1(3,1,3,512);
RX(3,1,2048);
RW0(4,1,0,4096);
RW1(4,1,0,4608);
RX(4,0,4096);
RW0(5,1,1,4096);
RW1(5,1,1,4608);
RX(5,0,6144);
MZ(0,0,0,0,0); SB; RW0(6,1,2,4096); SB;
MZ(0,1,0,0,1); SB; RW1(6,1,2,4608); RX(6,1,4096); MW(131072); SB;
MZ(0,0,1,1,0); SB; RW0(7,1,3,4096); DW(0,0,2); SB;
MZ(0,1,1,1,1); SB; RW1(7,1,3,4608); RX(7,1,6144); MW(132096); SB;
MZ(0,0,2,2,0); SB; RW0(0,1,0,8192); DW(1,0,2); SB;
MZ(0,1,2,2,1); SB; RW1(0,1,0,8704); MW(133120); SB;
MZ(0,0,3,3,0); SB; RW0(1,1,1,8192); DW(2,0,2); SB;
MZ(0,1,3,3,1); SB; RW1(1,1,1,8704); MW(134144); SB;
MFW(0,0,0,4,2); SB; RW0(2,1,2,8192); DW(3,0,2); SB;
MF(0,1,0,4,3); SB; RW1(2,1,2,8704); MX(65536); SB;
MF(0,0,1,5,2); SB; RW0(3,1,3,8192); DX(0,2); SB;
MF(0,1,1,5,3); SB; RW1(3,1,3,8704); MX(66560); SB;
MFW(0,0,2,6,2); SB; RW0(4,1,0,12288); DX(1,2); SB;
MF(0,1,2,6,3); SB; RW1(4,1,0,12800); MX(67584); SB;
MF(0,0,3,7,2); SB; RW0(5,1,1,12288); DX(2,2); SB;
MF(0,1,3,7,3); SB; RW1(5,1,1,12800); MX(68608); SB;
MFW(0,0,0,0,4); SB; RW0(6,1,2,12288); DX(3,2); SB;
MF(0,1,0,0,5); SB; RW1(6,1,2,12800); MX(69632); SB;
MF(0,0,1,1,4); SB; RW0(7,1,3,12288); DX(4,2); SB;
MF(0,1,1,1,5); SB; RW1(7,1,3,12800); SB;
WV(9); BAR;
MFW(0,0,2,2,4); SB; RW0(0,1,0,16384); SB;
MF(0,1,2,2,5); SB; RW1(0,1,0,16896); RX(8,0,32768); MX(70656); SB;
MF(0,0,3,3,4); SB; RW0(1,1,1,16384); DX(5,2); SB;
MF(0,1,3,3,5); SB; RW1(1,1,1,16896); RX(9,0,34816); MX(71680); SB;
MFW(0,0,0,4,6); SB; RW0(2,1,2,16384); DX(6,2); SB;
MF(0,1,0,4,7); SB; RW1(2,1,2,16896); RX(10,1,32768); MX(72704); SB;
MF(0,0,1,5,6); SB; RW0(3,1,3,16384); DX(7,2); SB;
MF(0,1,1,5,7); SB; RW1(3,1,3,16896); RX(11,1,34816); MW(147456); SB;
MFW(0,0,2,6,6); SB; RW0(4,1,0,20480); DW(0,0,3); SB;
MF(0,1,2,6,7); SB; RW1(4,1,0,20992); RX(12,0,36864); MW(148480); SB;
MF(0,0,3,7,6); SB; RW0(5,1,1,20480); DW(1,0,3); SB;
MF(0,1,3,7,7); SB; RW1(5,1,1,20992); RX(13,0,38912); MW(149504); SB;
MFW(0,0,0,0,8); SB; RW0(6,1,2,20480); DW(2,0,3); SB;
MF(0,1,0,0,9); SB; RW1(6,1,2,20992); RX(14,1,36864); MW(150528); SB;
MF(0,0,1,1,8); SB; RW0(7,1,3,20480); DW(3,0,3); SB;
MF(0,1,1,1,9); SB; RW1(7,1,3,20992); RX(15,1,38912); MX(0); SB;
MFW(0,0,2,2,8); SB; RW0(0,1,0,24576); DX(0,3); SB;
MF(0,1,2,2,9); SB; RW1(0,1,0,25088); MX(1024); SB;
MF(0,0,3,3,8); SB; RW0(1,1,1,24576); DX(1,3); SB;
MF(0,1,3,3,9); SB; RW1(1,1,1,25088); MX(2048); SB;
MFW(0,0,0,4,10); SB; RW0(2,1,2,24576); DX(2,3); SB;
MF(0,1,0,4,11); SB; RW1(2,1,2,25088); MX(3072); SB;
MF(0,0,1,5,10); SB; RW0(3,1,3,24576); DX(3,3); SB;
MF(0,1,1,5,11); SB; RW1(3,1,3,25088); MX(4096); SB;
MFW(0,0,2,6,10); SB; RW0(4,1,0,28672); DX(4,3); SB;
MF(0,1,2,6,11); SB; RW1(4,1,0,29184); MX(5120); SB;
MF(0,0,3,7,10); SB; RW0(5,1,1,28672); DX(5,3); SB;
MF(0,1,3,7,11); SB; RW1(5,1,1,29184); MX(6144); SB;
MFW(0,0,0,0,12); SB; RW0(6,1,2,28672); DX(6,3); SB;
MF(0,1,0,0,13); SB; RW1(6,1,2,29184); MX(7168); SB;
MF(0,0,1,1,12); SB; RW0(7,1,3,28672); DX(7,3); SB;
MF(0,1,1,1,13); SB; RW1(7,1,3,29184); SB;
WV(12); BAR;
MFW(0,0,2,2,12); SB; RW0(0,1,0,32768); SB;
MF(0,1,2,2,13); SB; RW1(0,1,0,33280); RX(16,0,65536); MW(98304); SB;
MF(0,0,3,3,12); SB; RW0(1,1,1,32768); DW(0,0,4); SB;
MF(0,1,3,3,13); SB; RW1(1,1,1,33280); RX(17,0,67584); MW(99328); SB;
MFW(0,0,0,4,14); SB; RW0(2,1,2,32768); DW(1,0,4); SB;
MF(0,1,0,4,15); SB; RW1(2,1,2,33280); RX(18,1,65536); MW(100352); SB;
MF(0,0,1,5,14); SB; RW0(3,1,3,32768); DW(2,0,4); SB;
MF(0,1,1,5,15); SB; RW1(3,1,3,33280); RX(19,1,67584); MW(101376); SB;
MFW(0,0,2,6,14); SB; RW0(4,1,0,36864); DW(3,0,4); SB;
MF(0,1,2,6,15); SB; RW1(4,1,0,37376); RX(20,0,69632); MX(32768); SB;
MF(0,0,3,7,14); SB; RW0(5,1,1,36864); DX(0,4); SB;
MF(0,1,3,7,15); SB; RW1(5,1,1,37376); RX(21,0,71680); MX(33792); SB;
MFW(0,0,0,0,16); SB; RW0(6,1,2,36864); DX(1,4); SB;
MF(0,1,0,0,17); SB; RW1(6,1,2,37376); RX(22,1,69632); MX(34816); SB;
MF(0,0,1,1,16); SB; RW0(7,1,3,36864); DX(2,4); SB;
MF(0,1,1,1,17); SB; RW1(7,1,3,37376); RX(23,1,71680); MX(35840); SB;
MFW(0,0,2,2,16); SB; RW0(0,1,0,40960); DX(3,4); SB;
MF(0,1,2,2,17); SB; RW1(0,1,0,41472); MX(36864); SB;
MF(0,0,3,3,16); SB; RW0(1,1,1,40960); DX(4,4); SB;
MF(0,1,3,3,17); SB; RW1(1,1,1,41472); MX(37888); SB;
MFW(0,0,0,4,18); SB; RW0(2,1,2,40960); DX(5,4); SB;
MF(0,1,0,4,19); SB; RW1(2,1,2,41472); MX(38912); SB;
MF(0,0,1,5,18); SB; RW0(3,1,3,40960); DX(6,4); SB;
MF(0,1,1,5,19); SB; RW1(3,1,3,41472); MX(39936); SB;
MFW(0,0,2,6,18); SB; RW0(4,1,0,45056); DX(7,4); SB;
MF(0,1,2,6,19); SB; RW1(4,1,0,45568); SB;
MF(0,0,3,7,18); SB; RW0(5,1,1,45056); SB;
MF(0,1,3,7,19); SB; RW1(5,1,1,45568); SB;
MFW(0,0,0,0,20); SB; RW0(6,1,2,45056); SB;
MF(0,1,0,0,21); SB; RW1(6,1,2,45568); SB;
MF(0,0,1,1,20); SB; RW0(7,1,3,45056); SB;
MF(0,1,1,1,21); SB; RW1(7,1,3,45568); SB;
WV(12); BAR;
MFW(0,0,2,2,20); SB; RW0(0,1,0,49152); SB;
MF(0,1,2,2,21); SB; RW1(0,1,0,49664); RX(24,0,0); MW(114688); SB;
MF(0,0,3,3,20); SB; RW0(1,1,1,49152); DW(0,0,5); SB;
MF(0,1,3,3,21); SB; RW1(1,1,1,49664); RX(25,0,2048); MW(115712); SB;
MFW(0,0,0,4,22); SB; RW0(2,1,2,49152); DW(1,0,5); SB;
MF(0,1,0,4,23); SB; RW1(2,1,2,49664); RX(26,1,0); MW(116736); SB;
MF(0,0,1,5,22); SB; RW0(3,1,3,49152); DW(2,0,5); SB;
MF(0,1,1,5,23); SB; RW1(3,1,3,49664); RX(27,1,2048); MW(117760); SB;
MFW(0,0,2,6,22); SB; RW0(4,1,0,53248); DW(3,0,5); SB;
MF(0,1,2,6,23); SB; RW1(4,1,0,53760); RX(28,0,4096); MX(65536); SB;
MF(0,0,3,7,22); SB; RW0(5,1,1,53248); DX(0,5); SB;
MF(0,1,3,7,23); SB; RW1(5,1,1,53760); RX(29,0,6144); MX(66560); SB;
MFW(0,0,0,0,24); SB; RW0(6,1,2,53248); DX(1,5); SB;
MF(0,1,0,0,25); SB; RW1(6,1,2,53760); RX(30,1,4096); MX(67584); SB;
MF(0,0,1,1,24); SB; RW0(7,1,3,53248); DX(2,5); SB;
MF(0,1,1,1,25); SB; RW1(7,1,3,53760); RX(31,1,6144); MX(68608); SB;
MFW(0,0,2,2,24); SB; RW0(0,1,0,57344); DX(3,5); SB;
MF(0,1,2,2,25); SB; RW1(0,1,0,57856); MX(69632); SB;
MF(0,0,3,3,24); SB; RW0(1,1,1,57344); DX(4,5); SB;
MF(0,1,3,3,25); SB; RW1(1,1,1,57856); MX(70656); SB;
MFW(0,0,0,4,26); SB; RW0(2,1,2,57344); DX(5,5); SB;
MF(0,1,0,4,27); SB; RW1(2,1,2,57856); MX(71680); SB;
MF(0,0,1,5,26); SB; RW0(3,1,3,57344); DX(6,5); SB;
MF(0,1,1,5,27); SB; RW1(3,1,3,57856); MX(72704); SB;
MFW(0,0,2,6,26); SB; RW0(4,1,0,61440); DX(7,5); SB;
MF(0,1,2,6,27); SB; RW1(4,1,0,61952); SB;
MF(0,0,3,7,26); SB; RW0(5,1,1,61440); SB;
MF(0,1,3,7,27); SB; RW1(5,1,1,61952); SB;
MFW(0,0,0,0,28); SB; RW0(6,1,2,61440); SB;
MF(0,1,0,0,29); SB; RW1(6,1,2,61952); SB;
MF(0,0,1,1,28); SB; RW0(7,1,3,61440); SB;
MF(0,1,1,1,29); SB; RW1(7,1,3,61952); SB;
TOUCHW;
WV(12); BAR;
MFW(0,0,2,2,28); SB; RW0(0,1,0,0); SB;
MF(0,1,2,2,29); SB; RW1(0,1,0,512); RX(32,0,32768); MW(131072); SB;
MF(0,0,3,3,28); SB; RW0(1,1,1,0); DW(0,0,6); SB;
MF(0,1,3,3,29); SB; RW1(1,1,1,512); RX(33,0,34816); MW(132096); SB;
MFW(0,0,0,4,30); SB; RW0(2,1,2,0); DW(1,0,6); SB;
MF(0,1,0,4,31); SB; RW1(2,1,2,512); RX(34,1,32768); MW(133120); SB;
MF(0,0,1,5,30); SB; RW0(3,1,3,0); DW(2,0,6); SB;
MF(0,1,1,5,31); SB; RW1(3,1,3,512); RX(35,1,34816); MW(134144); SB;
MFW(0,0,2,6,30); SB; RW0(4,1,0,4096); DW(3,0,6); SB;
MF(0,1,2,6,31); SB; RW1(4,1,0,4608); RX(36,0,36864); MX(0); SB;
MF(0,0,3,7,30); SB; RW0(5,1,1,4096); DX(0,6); SB;
MF(0,1,3,7,31); SB; RW1(5,1,1,4608); RX(37,0,38912); MX(1024); SB;
MFW(0,0,0,0,32); SB; RW0(6,1,2,4096); DX(1,6); SB;
MF(0,1,0,0,33); SB; RW1(6,1,2,4608); RX(38,1,36864); MX(2048); SB;
MF(0,0,1,1,32); SB; RW0(7,1,3,4096); DX(2,6); SB;
MF(0,1,1,1,33); SB; RW1(7,1,3,4608); RX(39,1,38912); MX(3072); SB;
MFW(0,0,2,2,32); SB; RW0(0,1,0,8192); DX(3,6); SB;
MF(0,1,2,2,33); SB; RW1(0,1,0,8704); MX(4096); SB;
MF(0,0,3,3,32); SB; RW0(1,1,1,8192); DX(4,6); SB;
MF(0,1,3,3,33); SB; RW1(1,1,1,8704); MX(5120); SB;
MFW(0,0,0,4,34); SB; RW0(2,1,2,8192); DX(5,6); SB;
MF(0,1,0,4,35); SB; RW1(2,1,2,8704); MX(6144); SB;
MF(0,0,1,5,34); SB; RW0(3,1,3,8192); DX(6,6); SB;
MF(0,1,1,5,35); SB; RW1(3,1,3,8704); MX(7168); SB;
MFW(0,0,2,6,34); SB; RW0(4,1,0,12288); DX(7,6); SB;
MF(0,1,2,6,35); SB; RW1(4,1,0,12800); SB;
MF(0,0,3,7,34); SB; RW0(5,1,1,12288); SB;
MF(0,1,3,7,35); SB; RW1(5,1,1,12800); SB;
MFW(0,0,0,0,36); SB; RW0(6,1,2,12288); SB;
MF(0,1,0,0,37); SB; RW1(6,1,2,12800); SB;
MF(0,0,1,1,36); SB; RW0(7,1,3,12288); SB;
MF(0,1,1,1,37); SB; RW1(7,1,3,12800); SB;
WV(12); BAR;
MFW(0,0,2,2,36); SB; RW0(0,1,0,16384); SB;
MF(0,1,2,2,37); SB; RW1(0,1,0,16896); RX(40,0,65536); MW(147456); SB;
MF(0,0,3,3,36); SB; RW0(1,1,1,16384); DW(0,0,7); SB;
MF(0,1,3,3,37); SB; RW1(1,1,1,16896); RX(41,0,67584); MW(148480); SB;
MFW(0,0,0,4,38); SB; RW0(2,1,2,16384); DW(1,0,7); SB;
MF(0,1,0,4,39); SB; RW1(2,1,2,16896); RX(42,1,65536); MW(149504); SB;
MF(0,0,1,5,38); SB; RW0(3,1,3,16384); DW(2,0,7); SB;
MF(0,1,1,5,39); SB; RW1(3,1,3,16896); RX(43,1,67584); MW(150528); SB;
MFW(0,0,2,6,38); SB; RW0(4,1,0,20480); DW(3,0,7); SB;
MF(0,1,2,6,39); SB; RW1(4,1,0,20992); RX(44,0,69632); MX(32768); SB;
MF(0,0,3,7,38); SB; RW0(5,1,1,20480); DX(0,7); SB;
MF(0,1,3,7,39); SB; RW1(5,1,1,20992); RX(45,0,71680); MX(33792); SB;
MFW(0,0,0,0,40); SB; RW0(6,1,2,20480); DX(1,7); SB;
MF(0,1,0,0,41); SB; RW1(6,1,2,20992); RX(46,1,69632); MX(34816); SB;
MF(0,0,1,1,40); SB; RW0(7,1,3,20480); DX(2,7); SB;
MF(0,1,1,1,41); SB; RW1(7,1,3,20992); RX(47,1,71680); MX(35840); SB;
MFW(0,0,2,2,40); SB; RW0(0,1,0,24576); DX(3,7); SB;
MF(0,1,2,2,41); SB; RW1(0,1,0,25088); MX(36864); SB;
MF(0,0,3,3,40); SB; RW0(1,1,1,24576); DX(4,7); SB;
MF(0,1,3,3,41); SB; RW1(1,1,1,25088); MX(37888); SB;
MFW(0,0,0,4,42); SB; RW0(2,1,2,24576); DX(5,7); SB;
MF(0,1,0,4,43); SB; RW1(2,1,2,25088); MX(38912); SB;
MF(0,0,1,5,42); SB; RW0(3,1,3,24576); DX(6,7); SB;
MF(0,1,1,5,43); SB; RW1(3,1,3,25088); MX(39936); SB;
MFW(0,0,2,6,42); SB; RW0(4,1,0,28672); DX(7,7); SB;
MF(0,1,2,6,43); SB; RW1(4,1,0,29184); SB;
MF(0,0,3,7,42); SB; RW0(5,1,1,28672); SB;
MF(0,1,3,7,43); SB; RW1(5,1,1,29184); SB;
MFW(0,0,0,0,44); SB; RW0(6,1,2,28672); SB;
MF(0,1,0,0,45); SB; RW1(6,1,2,29184); SB;
MF(0,0,1,1,44); SB; RW0(7,1,3,28672); SB;
MF(0,1,1,1,45); SB; RW1(7,1,3,29184); SB;
WV(12); BAR;
MFW(0,0,2,2,44); SB; RW0(0,1,0,32768); SB;
MF(0,1,2,2,45); SB; RW1(0,1,0,33280); RX(48,0,0); MW(98304); SB;
MF(0,0,3,3,44); SB; RW0(1,1,1,32768); DW(0,1,0); SB;
MF(0,1,3,3,45); SB; RW1(1,1,1,33280); RX(49,0,2048); SB;
MFW(0,0,0,4,46); SB; RW0(2,1,2,32768); SB;
MF(0,1,0,4,47); SB; RW1(2,1,2,33280); RX(50,1,0); MW(99328); SB;
MF(0,0,1,5,46); SB; RW0(3,1,3,32768); DW(1,1,0); SB;
MF(0,1,1,5,47); SB; RW1(3,1,3,33280); RX(51,1,2048); SB;
MFW(0,0,2,6,46); SB; RW0(4,1,0,36864); SB;
MF(0,1,2,6,47); SB; RW1(4,1,0,37376); RX(52,0,4096); MW(100352); SB;
MF(0,0,3,7,46); SB; RW0(5,1,1,36864); DW(2,1,0); SB;
MF(0,1,3,7,47); SB; RW1(5,1,1,37376); RX(53,0,6144); SB;
MFW(0,0,0,0,48); SB; RW0(6,1,2,36864); SB;
MF(0,1,0,0,49); SB; RW1(6,1,2,37376); RX(54,1,4096); MW(101376); SB;
MF(0,0,1,1,48); SB; RW0(7,1,3,36864); DW(3,1,0); SB;
MF(0,1,1,1,49); SB; RW1(7,1,3,37376); RX(55,1,6144); SB;
MFW(0,0,2,2,48); SB; RW0(0,1,0,40960); SB;
MF(0,1,2,2,49); SB; RW1(0,1,0,41472); MW(81920); SB;
MF(0,0,3,3,48); SB; RW0(1,1,1,40960); DW(0,1,1); SB;
MF(0,1,3,3,49); SB; RW1(1,1,1,41472); SB;
MFW(0,0,0,4,50); SB; RW0(2,1,2,40960); SB;
MF(0,1,0,4,51); SB; RW1(2,1,2,41472); MW(82944); SB;
MF(0,0,1,5,50); SB; RW0(3,1,3,40960); DW(1,1,1); SB;
MF(0,1,1,5,51); SB; RW1(3,1,3,41472); SB;
MFW(0,0,2,6,50); SB; RW0(4,1,0,45056); SB;
MF(0,1,2,6,51); SB; RW1(4,1,0,45568); MW(83968); SB;
MF(0,0,3,7,50); SB; RW0(5,1,1,45056); DW(2,1,1); SB;
MF(0,1,3,7,51); SB; RW1(5,1,1,45568); SB;
MFW(0,0,0,0,52); SB; RW0(6,1,2,45056); SB;
MF(0,1,0,0,53); SB; RW1(6,1,2,45568); MW(84992); SB;
MF(0,0,1,1,52); SB; RW0(7,1,3,45056); DW(3,1,1); SB;
MF(0,1,1,1,53); SB; RW1(7,1,3,45568); SB;
WV(8); BAR;
MFW(0,0,2,2,52); SB; RW0(0,1,0,49152); SB;
MF(0,1,2,2,53); SB; RW1(0,1,0,49664); RX(56,0,32768); MW(114688); SB;
MF(0,0,3,3,52); SB; RW0(1,1,1,49152); DW(0,1,2); SB;
MF(0,1,3,3,53); SB; RW1(1,1,1,49664); RX(57,0,34816); SB;
MFW(0,0,0,4,54); SB; RW0(2,1,0,53248); SB;
MF(0,1,0,4,55); SB; RW1(2,1,0,53760); RX(58,1,32768); MW(115712); SB;
MF(0,0,1,5,54); SB; RW0(3,1,1,53248); DW(1,1,2); SB;
MF(0,1,1,5,55); SB; RW1(3,1,1,53760); RX(59,1,34816); SB;
MFW(0,0,2,6,54); SB; RW0(4,1,0,57344); SB;
MF(0,1,2,6,55); SB; RW1(4,1,0,57856); RX(60,0,36864); MW(116736); SB;
MF(0,0,3,7,54); SB; RW0(5,1,1,57344); DW(2,1,2); SB;
MF(0,1,3,7,55); SB; RW1(5,1,1,57856); RX(61,0,38912); SB;
MFW(0,0,0,0,56); SB; RW0(6,1,0,61440); SB;
MF(0,1,0,0,57); SB; RW1(6,1,0,61952); RX(62,1,36864); MW(117760); SB;
MF(0,0,1,1,56); SB; RW0(7,1,1,61440); DW(3,1,2); SB;
MF(0,1,1,1,57); SB; RW1(7,1,1,61952); RX(63,1,38912); SB;
MFW(0,0,0,2,58); SB; RW0(0,1,2,49152); SB;
MF(0,1,0,2,59); SB; RW1(0,1,2,49664); MW(65536); SB;
MF(0,0,1,3,58); SB; RW0(1,1,3,49152); DW(0,1,3); SB;
MF(0,1,1,3,59); SB; RW1(1,1,3,49664); SB;
MFW(0,0,0,4,60); SB; RW0(2,1,2,53248); SB;
MF(0,1,0,4,61); SB; RW1(2,1,2,53760); MW(66560); SB;
MF(0,0,1,5,60); SB; RW0(3,1,3,53248); DW(1,1,3); SB;
MF(0,1,1,5,61); SB; RW1(3,1,3,53760); SB;
MFW(0,0,0,6,62); SB; RW0(4,1,2,57344); SB;
MF(0,1,0,6,63); SB; RW1(4,1,2,57856); MW(67584); SB;
MF(0,0,1,7,62); SB; RW0(5,1,3,57344); DW(2,1,3); SB;
MF(0,1,1,7,63); SB; RW1(5,1,3,57856); SB;
MFW(0,0,2,0,56); SB; RW0(6,1,2,61440); SB;
MF(0,1,2,0,57); SB; RW1(6,1,2,61952); MW(68608); SB;
MF(0,0,3,1,56); SB; RW0(7,1,3,61440); DW(3,1,3); SB;
MF(0,1,3,1,57); SB; RW1(7,1,3,61952); SB;
WV(8); BAR;
MFW(0,0,2,2,58); SB; RW0(0,1,0,0); PK0(0,0,0); PK1(0,0,0); SB;
MF(0,1,2,2,59); SB; RW1(0,1,0,512); PK2(0,0,0); PK3(0,0,0); SB;
MF(0,0,3,3,58); SB; RW0(1,1,1,0); PK4(0,0,0); PK5(0,0,0); SB;
MF(0,1,3,3,59); SB; RW1(1,1,1,512); PK6(0,0,0); SB;
MFW(0,0,2,4,60); SB; RW0(2,1,2,0); PK0(0,1,0); PK1(0,1,0); SB;
MF(0,1,2,4,61); SB; RW1(2,1,2,512); PK2(0,1,0); PK3(0,1,0); SB;
MF(0,0,3,5,60); SB; RW0(3,1,3,0); PK4(0,1,0); PK5(0,1,0); SB;
MF(0,1,3,5,61); SB; RW1(3,1,3,512); PK6(0,1,0); SB;
MFW(0,0,2,6,62); SB; RW0(4,1,0,4096); SB;
MF(0,1,2,6,63); SB; RW1(4,1,0,4608); MW(131072); SB;
MF(0,0,3,7,62); SB; RW0(5,1,1,4096); DW(0,1,4); SB;
MF(0,1,3,7,63); SB; RW1(5,1,1,4608); SB;
MZ(1,0,0,0,0); SB; RW0(6,1,2,4096); SB;
MZ(1,1,0,0,1); SB; RW1(6,1,2,4608); MW(132096); SB;
MZ(1,0,1,1,0); SB; RW0(7,1,3,4096); DW(1,1,4); SB;
MZ(1,1,1,1,1); SB; RW1(7,1,3,4608); SB;
MZ(1,0,2,2,0); SB; RW0(0,1,0,8192); PK0(0,0,1); PK1(0,0,1); SB;
MZ(1,1,2,2,1); SB; RW1(0,1,0,8704); PK2(0,0,1); PK3(0,0,1); SB;
MZ(1,0,3,3,0); SB; RW0(1,1,1,8192); PK4(0,0,1); PK5(0,0,1); SB;
MZ(1,1,3,3,1); SB; RW1(1,1,1,8704); PK6(0,0,1); SB;
MFW(1,0,0,4,2); SB; RW0(2,1,2,8192); PK0(0,1,1); PK1(0,1,1); SB;
MF(1,1,0,4,3); SB; RW1(2,1,2,8704); PK2(0,1,1); PK3(0,1,1); SB;
MF(1,0,1,5,2); SB; RW0(3,1,3,8192); PK4(0,1,1); PK5(0,1,1); SB;
MF(1,1,1,5,3); SB; RW1(3,1,3,8704); PK6(0,1,1); SB;
MFW(1,0,2,6,2); SB; RW0(4,1,0,12288); SB;
MF(1,1,2,6,3); SB; RW1(4,1,0,12800); MW(133120); SB;
MF(1,0,3,7,2); SB; RW0(5,1,1,12288); DW(2,1,4); SB;
MF(1,1,3,7,3); SB; RW1(5,1,1,12800); SB;
MFW(1,0,0,0,4); SB; RW0(6,1,2,12288); SB;
MF(1,1,0,0,5); SB; RW1(6,1,2,12800); MW(134144); SB;
MF(1,0,1,1,4); SB; RW0(7,1,3,12288); DW(3,1,4); SB;
MF(1,1,1,1,5); SB; RW1(7,1,3,12800); SB;
MFW(1,0,2,2,4); SB; RW0(0,0,0,49152); SB;
MF(1,1,2,2,5); SB; RW1(0,0,0,49664); MW(49152); SB;
MF(1,0,3,3,4); SB; RW0(1,0,1,49152); DW(0,1,5); SB;
MF(1,1,3,3,5); SB; RW1(1,0,1,49664); SB;
MFW(1,0,0,4,6); SB; RW0(2,0,2,49152); SB;
MF(1,1,0,4,7); SB; RW1(2,0,2,49664); MW(50176); SB;
MF(1,0,1,5,6); SB; RW0(3,0,3,49152); DW(1,1,5); SB;
MF(1,1,1,5,7); SB; RW1(3,0,3,49664); SB;
BAR;
MFW(1,0,2,6,6); SB; RW0(4,0,0,53248); GO(0); SB;
MF(1,1,2,6,7); SB; RW1(4,0,0,53760); MW(51200); SB;
MF(1,0,3,7,6); SB; RW0(5,0,1,53248); DW(2,1,5); SB;
MF(1,1,3,7,7); SB; RW1(5,0,1,53760); SB;
MFW(1,0,0,0,8); SB; RW0(6,0,2,53248); SB;
MF(1,1,0,0,9); SB; RW1(6,0,2,53760); MW(52224); SB;
MF(1,0,1,1,8); SB; RW0(7,0,3,53248); DW(3,1,5); SB;
MF(1,1,1,1,9); SB; RW1(7,0,3,53760); SB;
MFW(1,0,2,2,8); SB; RW0(0,0,0,57344); SB;
MF(1,1,2,2,9); SB; RW1(0,0,0,57856); MW(147456); SB;
MF(1,0,3,3,8); SB; RW0(1,0,1,57344); DW(0,1,6); SB;
MF(1,1,3,3,9); SB; RW1(1,0,1,57856); SB;
MFW(1,0,0,4,10); SB; RW0(2,0,2,57344); SB;
MF(1,1,0,4,11); SB; RW1(2,0,2,57856); MW(148480); SB;
MF(1,0,1,5,10); SB; RW0(3,0,3,57344); DW(1,1,6); SB;
MF(1,1,1,5,11); SB; RW1(3,0,3,57856); SB;
MFW(1,0,2,6,10); SB; RW0(4,0,0,61440); SB;
MF(1,1,2,6,11); SB; RW1(4,0,0,61952); MW(149504); SB;
MF(1,0,3,7,10); SB; RW0(5,0,1,61440); DW(2,1,6); SB;
MF(1,1,3,7,11); SB; RW1(5,0,1,61952); SB;
MFW(1,0,0,0,12); SB; RW0(6,0,2,61440); SB;
MF(1,1,0,0,13); SB; RW1(6,0,2,61952); MW(150528); SB;
MF(1,0,1,1,12); SB; RW0(7,0,3,61440); DW(3,1,6); SB;
MF(1,1,1,1,13); SB; RW1(7,0,3,61952); SB;
WV(12); BAR;
MFW(1,0,2,2,12); SB; RW0(0,1,0,16384); SB;
MF(1,1,2,2,13); SB; RW1(0,1,0,16896); MW(98304); SB;
MF(1,0,3,3,12); SB; RW0(1,1,1,16384); DW(0,1,7); SB;
MF(1,1,3,3,13); SB; RW1(1,1,1,16896); SB;
MFW(1,0,0,4,14); SB; RW0(2,1,2,16384); SB;
MF(1,1,0,4,15); SB; RW1(2,1,2,16896); MW(99328); SB;
MF(1,0,1,5,14); SB; RW0(3,1,3,16384); DW(1,1,7); SB;
MF(1,1,1,5,15); SB; RW1(3,1,3,16896); SB;
MFW(1,0,2,6,14); SB; RW0(4,1,0,20480); SB;
MF(1,1,2,6,15); SB; RW1(4,1,0,20992); MW(100352); SB;
MF(1,0,3,7,14); SB; RW0(5,1,1,20480); DW(2,1,7); SB;
MF(1,1,3,7,15); SB; RW1(5,1,1,20992); SB;
MFW(1,0,0,0,16); SB; RW0(6,1,2,20480); SB;
MF(1,1,0,0,17); SB; RW1(6,1,2,20992); MW(101376); SB;
MF(1,0,1,1,16); SB; RW0(7,1,3,20480); DW(3,1,7); SB;
MF(1,1,1,1,17); SB; RW1(7,1,3,20992); SB;
MFW(1,0,2,2,16); SB; RW0(0,1,0,24576); SB;
MF(1,1,2,2,17); SB; RW1(0,1,0,25088); MW(32768); SB;
MF(1,0,3,3,16); SB; RW0(1,1,1,24576); DW(0,2,0); SB;
MF(1,1,3,3,17); SB; RW1(1,1,1,25088); SB;
MFW(1,0,0,4,18); SB; RW0(2,1,2,24576); SB;
MF(1,1,0,4,19); SB; RW1(2,1,2,25088); MW(33792); SB;
MF(1,0,1,5,18); SB; RW0(3,1,3,24576); DW(1,2,0); SB;
MF(1,1,1,5,19); SB; RW1(3,1,3,25088); SB;
MFW(1,0,2,6,18); SB; RW0(4,1,0,28672); SB;
MF(1,1,2,6,19); SB; RW1(4,1,0,29184); MW(34816); SB;
MF(1,0,3,7,18); SB; RW0(5,1,1,28672); DW(2,2,0); SB;
MF(1,1,3,7,19); SB; RW1(5,1,1,29184); SB;
MFW(1,0,0,0,20); SB; RW0(6,1,2,28672); SB;
MF(1,1,0,0,21); SB; RW1(6,1,2,29184); MW(35840); SB;
MF(1,0,1,1,20); SB; RW0(7,1,3,28672); DW(3,2,0); SB;
MF(1,1,1,1,21); SB; RW1(7,1,3,29184); SB;
MFW(1,0,2,2,20); SB; RW0(0,0,0,32768); SB;
MF(1,1,2,2,21); SB; RW1(0,0,0,33280); SB;
MF(1,0,3,3,20); SB; RW0(1,0,1,32768); SB;
MF(1,1,3,3,21); SB; RW1(1,0,1,33280); SB;
MFW(1,0,0,4,22); SB; RW0(2,0,2,32768); SB;
MF(1,1,0,4,23); SB; RW1(2,0,2,33280); SB;
MF(1,0,1,5,22); SB; RW0(3,0,3,32768); SB;
MF(1,1,1,5,23); SB; RW1(3,0,3,33280); SB;
MFW(1,0,2,6,22); SB; RW0(4,0,0,36864); GL(0,0); SB;
MF(1,1,2,6,23); SB; RW1(4,0,0,37376); GL(0,1); SB;
MF(1,0,3,7,22); SB; RW0(5,0,1,36864); GL(0,2); SB;
MF(1,1,3,7,23); SB; RW1(5,0,1,37376); SB;
MFW(1,0,0,0,24); SB; RW0(6,0,2,36864); SB;
MF(1,1,0,0,25); SB; RW1(6,0,2,37376); SB;
MF(1,0,1,1,24); SB; RW0(7,0,3,36864); SB;
MF(1,1,1,1,25); SB; RW1(7,0,3,37376); SB;
MFW(1,0,2,2,24); SB; RW0(0,0,0,40960); SB;
MF(1,1,2,2,25); SB; RW1(0,0,0,41472); SB;
MF(1,0,3,3,24); SB; RW0(1,0,1,40960); SB;
MF(1,1,3,3,25); SB; RW1(1,0,1,41472); SB;
MFW(1,0,0,4,26); SB; RW0(2,0,2,40960); SB;
MF(1,1,0,4,27); SB; RW1(2,0,2,41472); SB;
MF(1,0,1,5,26); SB; RW0(3,0,3,40960); SB;
MF(1,1,1,5,27); SB; RW1(3,0,3,41472); SB;
MFW(1,0,2,6,26); SB; RW0(4,0,0,45056); SB;
MF(1,1,2,6,27); SB; RW1(4,0,0,45568); SB;
MF(1,0,3,7,26); SB; RW0(5,0,1,45056); SB;
MF(1,1,3,7,27); SB; RW1(5,0,1,45568); SB;
MFW(1,0,0,0,28); SB; RW0(6,0,2,45056); SB;
MF(1,1,0,0,29); SB; RW1(6,0,2,45568); SB;
MF(1,0,1,1,28); SB; RW0(7,0,3,45056); SB;
MF(1,1,1,1,29); SB; RW1(7,0,3,45568); SB;
WV(15); BAR;
MFW(1,0,2,2,28); SB; RW0(0,1,0,32768); SB;
MF(1,1,2,2,29); SB; RW1(0,1,0,33280); MW(114688); SB;
MF(1,0,3,3,28); SB; RW0(1,1,1,32768); DW(0,2,1); SB;
MF(1,1,3,3,29); SB; RW1(1,1,1,33280); SB;
MFW(1,0,0,4,30); SB; RW0(2,1,2,32768); SB;
MF(1,1,0,4,31); SB; RW1(2,1,2,33280); MW(115712); SB;
MF(1,0,1,5,30); SB; RW0(3,1,3,32768); DW(1,2,1); SB;
MF(1,1,1,5,31); SB; RW1(3,1,3,33280); SB;
MFW(1,0,2,6,30); SB; RW0(4,1,0,36864); SB;
MF(1,1,2,6,31); SB; RW1(4,1,0,37376); MW(116736); SB;
MF(1,0,3,7,30); SB; RW0(5,1,1,36864); DW(2,2,1); SB;
MF(1,1,3,7,31); SB; RW1(5,1,1,37376); SB;
MFW(1,0,0,0,32); SB; RW0(6,1,2,36864); SB;
MF(1,1,0,0,33); SB; RW1(6,1,2,37376); MW(117760); SB;
MF(1,0,1,1,32); SB; RW0(7,1,3,36864); DW(3,2,1); SB;
MF(1,1,1,1,33); SB; RW1(7,1,3,37376); SB;
MFW(1,0,2,2,32); SB; RW0(0,1,0,40960); SB;
MF(1,1,2,2,33); SB; RW1(0,1,0,41472); MW(81920); SB;
MF(1,0,3,3,32); SB; RW0(1,1,1,40960); DW(0,2,2); SB;
MF(1,1,3,3,33); SB; RW1(1,1,1,41472); SB;
MFW(1,0,0,4,34); SB; RW0(2,1,2,40960); SB;
MF(1,1,0,4,35); SB; RW1(2,1,2,41472); MW(82944); SB;
MF(1,0,1,5,34); SB; RW0(3,1,3,40960); DW(1,2,2); SB;
MF(1,1,1,5,35); SB; RW1(3,1,3,41472); SB;
MFW(1,0,2,6,34); SB; RW0(4,1,0,45056); SB;
MF(1,1,2,6,35); SB; RW1(4,1,0,45568); MW(83968); SB;
MF(1,0,3,7,34); SB; RW0(5,1,1,45056); DW(2,2,2); SB;
MF(1,1,3,7,35); SB; RW1(5,1,1,45568); SB;
MFW(1,0,0,0,36); SB; RW0(6,1,2,45056); SB;
MF(1,1,0,0,37); SB; RW1(6,1,2,45568); MW(84992); SB;
MF(1,0,1,1,36); SB; RW0(7,1,3,45056); DW(3,2,2); SB;
MF(1,1,1,1,37); SB; RW1(7,1,3,45568); SB;
MFW(1,0,2,2,36); SB; RW0(0,0,0,16384); GW(8); GK0; SB;
MF(1,1,2,2,37); SB; RW1(0,0,0,16896); GK1; GC(0); GM(0,0); SB;
MF(1,0,3,3,36); SB; RW0(1,0,1,16384); GM(1,0); SB;
MF(1,1,3,3,37); SB; RW1(1,0,1,16896); GM(2,0); SB;
MFW(1,0,0,4,38); SB; RW0(2,0,2,16384); GM(3,0); SB;
MF(1,1,0,4,39); SB; RW1(2,0,2,16896); SB;
MF(1,0,1,5,38); SB; RW0(3,0,3,16384); SB;
MF(1,1,1,5,39); SB; RW1(3,0,3,16896); SB;
MFW(1,0,2,6,38); SB; RW0(4,0,0,20480); SB;
MF(1,1,2,6,39); SB; RW1(4,0,0,20992); SB;
MF(1,0,3,7,38); SB; RW0(5,0,1,20480); SB;
MF(1,1,3,7,39); SB; RW1(5,0,1,20992); SB;
MFW(1,0,0,0,40); SB; RW0(6,0,2,20480); GP(0); GP(1); SB;
MF(1,1,0,0,41); SB; RW1(6,0,2,20992); GP(2); GP(3); SB;
MF(1,0,1,1,40); SB; RW0(7,0,3,20480); GT(0); SB;
MF(1,1,1,1,41); SB; RW1(7,0,3,20992); SB;
MFW(1,0,2,2,40); SB; RW0(0,0,0,24576); SB;
MF(1,1,2,2,41); SB; RW1(0,0,0,25088); SB;
MF(1,0,3,3,40); SB; RW0(1,0,1,24576); SB;
MF(1,1,3,3,41); SB; RW1(1,0,1,25088); SB;
MFW(1,0,0,4,42); SB; RW0(2,0,2,24576); SB;
MF(1,1,0,4,43); SB; RW1(2,0,2,25088); SB;
MF(1,0,1,5,42); SB; RW0(3,0,3,24576); SB;
MF(1,1,1,5,43); SB; RW1(3,0,3,25088); SB;
MFW(1,0,2,6,42); SB; RW0(4,0,0,28672); SB;
MF(1,1,2,6,43); SB; RW1(4,0,0,29184); SB;
MF(1,0,3,7,42); SB; RW0(5,0,1,28672); SB;
MF(1,1,3,7,43); SB; RW1(5,0,1,29184); SB;
MFW(1,0,0,0,44); SB; RW0(6,0,2,28672); SB;
MF(1,1,0,0,45); SB; RW1(6,0,2,29184); SB;
MF(1,0,1,1,44); SB; RW0(7,0,3,28672); SB;
MF(1,1,1,1,45); SB; RW1(7,0,3,29184); SB;
WV(16); BAR;
MFW(1,0,2,2,44); SB; RW0(0,1,0,49152); SB;
MF(1,1,2,2,45); SB; RW1(0,1,0,49664); MW(131072); SB;
MF(1,0,3,3,44); SB; RW0(1,1,1,49152); DW(0,2,3); SB;
MF(1,1,3,3,45); SB; RW1(1,1,1,49664); SB;
MFW(1,0,0,4,46); SB; RW0(2,1,2,49152); SB;
MF(1,1,0,4,47); SB; RW1(2,1,2,49664); MW(132096); SB;
MF(1,0,1,5,46); SB; RW0(3,1,3,49152); DW(1,2,3); SB;
MF(1,1,1,5,47); SB; RW1(3,1,3,49664); SB;
MFW(1,0,2,6,46); SB; RW0(4,1,0,53248); SB;
MF(1,1,2,6,47); SB; RW1(4,1,0,53760); MW(133120); SB;
MF(1,0,3,7,46); SB; RW0(5,1,1,53248); DW(2,2,3); SB;
MF(1,1,3,7,47); SB; RW1(5,1,1,53760); SB;
MFW(1,0,0,0,48); SB; RW0(6,1,2,53248); SB;
MF(1,1,0,0,49); SB; RW1(6,1,2,53760); MW(134144); SB;
MF(1,0,1,1,48); SB; RW0(7,1,3,53248); DW(3,2,3); SB;
MF(1,1,1,1,49); SB; RW1(7,1,3,53760); SB;
MFW(1,0,2,2,48); SB; RW0(0,1,0,57344); SB;
MF(1,1,2,2,49); SB; RW1(0,1,0,57856); MW(65536); SB;
MF(1,0,3,3,48); SB; RW0(1,1,1,57344); DW(0,2,4); SB;
MF(1,1,3,3,49); SB; RW1(1,1,1,57856); SB;
MFW(1,0,0,4,50); SB; RW0(2,1,2,57344); SB;
MF(1,1,0,4,51); SB; RW1(2,1,2,57856); MW(66560); SB;
MF(1,0,1,5,50); SB; RW0(3,1,3,57344); DW(1,2,4); SB;
MF(1,1,1,5,51); SB; RW1(3,1,3,57856); SB;
MFW(1,0,2,6,50); SB; RW0(4,1,0,61440); SB;
MF(1,1,2,6,51); SB; RW1(4,1,0,61952); MW(67584); SB;
MF(1,0,3,7,50); SB; RW0(5,1,1,61440); DW(2,2,4); SB;
MF(1,1,3,7,51); SB; RW1(5,1,1,61952); SB;
MFW(1,0,0,0,52); SB; RW0(6,1,2,61440); SB;
MF(1,1,0,0,53); SB; RW1(6,1,2,61952); MW(68608); SB;
MF(1,0,1,1,52); SB; RW0(7,1,3,61440); DW(3,2,4); SB;
MF(1,1,1,1,53); SB; RW1(7,1,3,61952); SB;
MFW(1,0,2,2,52); SB; RW0(0,1,0,0); SB;
MF(1,1,2,2,53); SB; RW1(0,1,0,512); SB;
MF(1,0,3,3,52); SB; RW0(1,1,1,0); SB;
MF(1,1,3,3,53); SB; RW1(1,1,1,512); SB;
MFW(1,0,0,4,54); SB; RW0(2,1,0,4096); SB;
MF(1,1,0,4,55); SB; RW1(2,1,0,4608); SB;
MF(1,0,1,5,54); SB; RW0(3,1,1,4096); SB;
MF(1,1,1,5,55); SB; RW1(3,1,1,4608); SB;
MFW(1,0,2,6,54); SB; RW0(4,1,0,8192); SB;
MF(1,1,2,6,55); SB; RW1(4,1,0,8704); SB;
MF(1,0,3,7,54); SB; RW0(5,1,1,8192); SB;
MF(1,1,3,7,55); SB; RW1(5,1,1,8704); SB;
MFW(1,0,0,0,56); SB; RW0(6,1,0,12288); SB;
MF(1,1,0,0,57); SB; RW1(6,1,0,12800); SB;
MF(1,0,1,1,56); SB; RW0(7,1,1,12288); SB;
MF(1,1,1,1,57); SB; RW1(7,1,1,12800); SB;
MFW(1,0,0,2,58); SB; RW0(0,1,2,0); SB;
MF(1,1,0,2,59); SB; RW1(0,1,2,512); SB;
MF(1,0,1,3,58); SB; RW0(1,1,3,0); SB;
MF(1,1,1,3,59); SB; RW1(1,1,3,512); SB;
MFW(1,0,0,4,60); SB; RW0(2,1,2,4096); SB;
MF(1,1,0,4,61); SB; RW1(2,1,2,4608); SB;
MF(1,0,1,5,60); SB; RW0(3,1,3,4096); SB;
MF(1,1,1,5,61); SB; RW1(3,1,3,4608); SB;
MFW(1,0,0,6,62); SB; RW0(4,1,2,8192); SB;
MF(1,1,0,6,63); SB; RW1(4,1,2,8704); SB;
MF(1,0,1,7,62); SB; RW0(5,1,3,8192); SB;
MF(1,1,1,7,63); SB; RW1(5,1,3,8704); SB;
MFW(1,0,2,0,56); SB; RW0(6,1,2,12288); SB;
MF(1,1,2,0,57); SB; RW1(6,1,2,12800); SB;
MF(1,0,3,1,56); SB; RW0(7,1,3,12288); SB;
MF(1,1,3,1,57); SB; RW1(7,1,3,12800); SB;
WV(13); BAR;
MFW(1,0,2,2,58); SB; RW0(0,0,0,0); PK0(1,0,0); PK1(1,0,0); SB;
MF(1,1,2,2,59); SB; RW1(0,0,0,512); PK2(1,0,0); PK3(1,0,0); SB;
MF(1,0,3,3,58); SB; RW0(1,0,1,0); PK4(1,0,0); PK5(1,0,0); SB;
MF(1,1,3,3,59); SB; RW1(1,0,1,512); PK6(1,0,0); SB;
MFW(1,0,2,4,60); SB; RW0(2,0,2,0); PK0(1,1,0); PK1(1,1,0); SB;
MF(1,1,2,4,61); SB; RW1(2,0,2,512); PK2(1,1,0); PK3(1,1,0); SB;
MF(1,0,3,5,60); SB; RW0(3,0,3,0); PK4(1,1,0); PK5(1,1,0); SB;
MF(1,1,3,5,61); SB; RW1(3,0,3,512); PK6(1,1,0); SB;
MFW(1,0,2,6,62); SB; RW0(4,0,0,4096); SB;
MF(1,1,2,6,63); SB; RW1(4,0,0,4608); MW(147456); SB;
MF(1,0,3,7,62); SB; RW0(5,0,1,4096); DW(0,2,5); SB;
MF(1,1,3,7,63); SB; RW1(5,0,1,4608); SB;
MZ(2,0,0,0,0); SB; RW0(6,0,2,4096); SB;
MZ(2,1,0,0,1); SB; RW1(6,0,2,4608); MW(148480); SB;
MZ(2,0,1,1,0); SB; RW0(7,0,3,4096); DW(1,2,5); SB;
MZ(2,1,1,1,1); SB; RW1(7,0,3,4608); SB;
MZ(2,0,2,2,0); SB; RW0(0,0,0,8192); PK0(1,0,1); PK1(1,0,1); SB;
MZ(2,1,2,2,1); SB; RW1(0,0,0,8704); PK2(1,0,1); PK3(1,0,1); SB;
MZ(2,0,3,3,0); SB; RW0(1,0,1,8192); PK4(1,0,1); PK5(1,0,1); SB;
MZ(2,1,3,3,1); SB; RW1(1,0,1,8704); PK6(1,0,1); SB;
MFW(2,0,0,4,2); SB; RW0(2,0,2,8192); PK0(1,1,1); PK1(1,1,1); SB;
MF(2,1,0,4,3); SB; RW1(2,0,2,8704); PK2(1,1,1); PK3(1,1,1); SB;
MF(2,0,1,5,2); SB; RW0(3,0,3,8192); PK4(1,1,1); PK5(1,1,1); SB;
MF(2,1,1,5,3); SB; RW1(3,0,3,8704); PK6(1,1,1); SB;
MFW(2,0,2,6,2); SB; RW0(4,0,0,12288); SB;
MF(2,1,2,6,3); SB; RW1(4,0,0,12800); MW(149504); SB;
MF(2,0,3,7,2); SB; RW0(5,0,1,12288); DW(2,2,5); SB;
MF(2,1,3,7,3); SB; RW1(5,0,1,12800); SB;
MFW(2,0,0,0,4); SB; RW0(6,0,2,12288); SB;
MF(2,1,0,0,5); SB; RW1(6,0,2,12800); MW(150528); SB;
MF(2,0,1,1,4); SB; RW0(7,0,3,12288); DW(3,2,5); SB;
MF(2,1,1,1,5); SB; RW1(7,0,3,12800); SB;
MFW(2,0,2,2,4); SB; RW0(0,1,0,16384); SB;
MF(2,1,2,2,5); SB; RW1(0,1,0,16896); MW(49152); SB;
MF(2,0,3,3,4); SB; RW0(1,1,1,16384); DW(0,2,6); SB;
MF(2,1,3,3,5); SB; RW1(1,1,1,16896); SB;
MFW(2,0,0,4,6); SB; RW0(2,1,2,16384); SB;
MF(2,1,0,4,7); SB; RW1(2,1,2,16896); MW(50176); SB;
MF(2,0,1,5,6); SB; RW0(3,1,3,16384); DW(1,2,6); SB;
MF(2,1,1,5,7); SB; RW1(3,1,3,16896); SB;
BAR;
MFW(2,0,2,6,6); SB; RW0(4,1,0,20480); GO(1); SB;
MF(2,1,2,6,7); SB; RW1(4,1,0,20992); MW(51200); SB;
MF(2,0,3,7,6); SB; RW0(5,1,1,20480); DW(2,2,6); SB;
MF(2,1,3,7,7); SB; RW1(5,1,1,20992); SB;
MFW(2,0,0,0,8); SB; RW0(6,1,2,20480); SB;
MF(2,1,0,0,9); SB; RW1(6,1,2,20992); MW(52224); SB;
MF(2,0,1,1,8); SB; RW0(7,1,3,20480); DW(3,2,6); SB;
MF(2,1,1,1,9); SB; RW1(7,1,3,20992); SB;
MFW(2,0,2,2,8); SB; RW0(0,1,0,24576); SB;
MF(2,1,2,2,9); SB; RW1(0,1,0,25088); MW(98304); SB;
MF(2,0,3,3,8); SB; RW0(1,1,1,24576); DW(0,2,7); SB;
MF(2,1,3,3,9); SB; RW1(1,1,1,25088); SB;
MFW(2,0,0,4,10); SB; RW0(2,1,2,24576); SB;
MF(2,1,0,4,11); SB; RW1(2,1,2,25088); MW(99328); SB;
MF(2,0,1,5,10); SB; RW0(3,1,3,24576); DW(1,2,7); SB;
MF(2,1,1,5,11); SB; RW1(3,1,3,25088); SB;
MFW(2,0,2,6,10); SB; RW0(4,1,0,28672); SB;
MF(2,1,2,6,11); SB; RW1(4,1,0,29184); MW(100352); SB;
MF(2,0,3,7,10); SB; RW0(5,1,1,28672); DW(2,2,7); SB;
MF(2,1,3,7,11); SB; RW1(5,1,1,29184); SB;
MFW(2,0,0,0,12); SB; RW0(6,1,2,28672); SB;
MF(2,1,0,0,13); SB; RW1(6,1,2,29184); MW(101376); SB;
MF(2,0,1,1,12); SB; RW0(7,1,3,28672); DW(3,2,7); SB;
MF(2,1,1,1,13); SB; RW1(7,1,3,29184); SB;
WV(16); BAR;
MFW(2,0,2,2,12); SB; RW0(0,0,0,49152); SB;
MF(2,1,2,2,13); SB; RW1(0,0,0,49664); MW(32768); SB;
MF(2,0,3,3,12); SB; RW0(1,0,1,49152); DW(0,3,0); SB;
MF(2,1,3,3,13); SB; RW1(1,0,1,49664); SB;
MFW(2,0,0,4,14); SB; RW0(2,0,2,49152); SB;
MF(2,1,0,4,15); SB; RW1(2,0,2,49664); MW(33792); SB;
MF(2,0,1,5,14); SB; RW0(3,0,3,49152); DW(1,3,0); SB;
MF(2,1,1,5,15); SB; RW1(3,0,3,49664); SB;
MFW(2,0,2,6,14); SB; RW0(4,0,0,53248); SB;
MF(2,1,2,6,15); SB; RW1(4,0,0,53760); MW(34816); SB;
MF(2,0,3,7,14); SB; RW0(5,0,1,53248); DW(2,3,0); SB;
MF(2,1,3,7,15); SB; RW1(5,0,1,53760); SB;
MFW(2,0,0,0,16); SB; RW0(6,0,2,53248); SB;
MF(2,1,0,0,17); SB; RW1(6,0,2,53760); MW(35840); SB;
MF(2,0,1,1,16); SB; RW0(7,0,3,53248); DW(3,3,0); SB;
MF(2,1,1,1,17); SB; RW1(7,0,3,53760); SB;
MFW(2,0,2,2,16); SB; RW0(0,0,0,57344); SB;
MF(2,1,2,2,17); SB; RW1(0,0,0,57856); SB;
MF(2,0,3,3,16); SB; RW0(1,0,1,57344); SB;
MF(2,1,3,3,17); SB; RW1(1,0,1,57856); SB;
MFW(2,0,0,4,18); SB; RW0(2,0,2,57344); SB;
MF(2,1,0,4,19); SB; RW1(2,0,2,57856); SB;
MF(2,0,1,5,18); SB; RW0(3,0,3,57344); SB;
MF(2,1,1,5,19); SB; RW1(3,0,3,57856); SB;
MFW(2,0,2,6,18); SB; RW0(4,0,0,61440); SB;
MF(2,1,2,6,19); SB; RW1(4,0,0,61952); SB;
MF(2,0,3,7,18); SB; RW0(5,0,1,61440); SB;
MF(2,1,3,7,19); SB; RW1(5,0,1,61952); SB;
MFW(2,0,0,0,20); SB; RW0(6,0,2,61440); SB;
MF(2,1,0,0,21); SB; RW1(6,0,2,61952); SB;
MF(2,0,1,1,20); SB; RW0(7,0,3,61440); SB;
MF(2,1,1,1,21); SB; RW1(7,0,3,61952); SB;
MFW(2,0,2,2,20); SB; RW0(0,1,0,32768); SB;
MF(2,1,2,2,21); SB; RW1(0,1,0,33280); SB;
MF(2,0,3,3,20); SB; RW0(1,1,1,32768); SB;
MF(2,1,3,3,21); SB; RW1(1,1,1,33280); SB;
MFW(2,0,0,4,22); SB; RW0(2,1,2,32768); SB;
MF(2,1,0,4,23); SB; RW1(2,1,2,33280); SB;
MF(2,0,1,5,22); SB; RW0(3,1,3,32768); SB;
MF(2,1,1,5,23); SB; RW1(3,1,3,33280); SB;
MFW(2,0,2,6,22); SB; RW0(4,1,0,36864); GL(1,0); SB;
MF(2,1,2,6,23); SB; RW1(4,1,0,37376); GL(1,1); SB;
MF(2,0,3,7,22); SB; RW0(5,1,1,36864); GL(1,2); SB;
MF(2,1,3,7,23); SB; RW1(5,1,1,37376); SB;
MFW(2,0,0,0,24); SB; RW0(6,1,2,36864); SB;
MF(2,1,0,0,25); SB; RW1(6,1,2,37376); SB;
MF(2,0,1,1,24); SB; RW0(7,1,3,36864); SB;
MF(2,1,1,1,25); SB; RW1(7,1,3,37376); SB;
MFW(2,0,2,2,24); SB; RW0(0,1,0,40960); SB;
MF(2,1,2,2,25); SB; RW1(0,1,0,41472); SB;
MF(2,0,3,3,24); SB; RW0(1,1,1,40960); SB;
MF(2,1,3,3,25); SB; RW1(1,1,1,41472); SB;
MFW(2,0,0,4,26); SB; RW0(2,1,2,40960); SB;
MF(2,1,0,4,27); SB; RW1(2,1,2,41472); SB;
MF(2,0,1,5,26); SB; RW0(3,1,3,40960); SB;
MF(2,1,1,5,27); SB; RW1(3,1,3,41472); SB;
MFW(2,0,2,6,26); SB; RW0(4,1,0,45056); SB;
MF(2,1,2,6,27); SB; RW1(4,1,0,45568); SB;
MF(2,0,3,7,26); SB; RW0(5,1,1,45056); SB;
MF(2,1,3,7,27); SB; RW1(5,1,1,45568); SB;
MFW(2,0,0,0,28); SB; RW0(6,1,2,45056); SB;
MF(2,1,0,0,29); SB; RW1(6,1,2,45568); SB;
MF(2,0,1,1,28); SB; RW0(7,1,3,45056); SB;
MF(2,1,1,1,29); SB; RW1(7,1,3,45568); SB;
WV(15); BAR;
MFW(2,0,2,2,28); SB; RW0(0,0,0,32768); SB;
MF(2,1,2,2,29); SB; RW1(0,0,0,33280); MW(114688); SB;
MF(2,0,3,3,28); SB; RW0(1,0,1,32768); DW(0,3,1); SB;
MF(2,1,3,3,29); SB; RW1(1,0,1,33280); SB;
MFW(2,0,0,4,30); SB; RW0(2,0,2,32768); SB;
MF(2,1,0,4,31); SB; RW1(2,0,2,33280); MW(115712); SB;
MF(2,0,1,5,30); SB; RW0(3,0,3,32768); DW(1,3,1); SB;
MF(2,1,1,5,31); SB; RW1(3,0,3,33280); SB;
MFW(2,0,2,6,30); SB; RW0(4,0,0,36864); SB;
MF(2,1,2,6,31); SB; RW1(4,0,0,37376); MW(116736); SB;
MF(2,0,3,7,30); SB; RW0(5,0,1,36864); DW(2,3,1); SB;
MF(2,1,3,7,31); SB; RW1(5,0,1,37376); SB;
MFW(2,0,0,0,32); SB; RW0(6,0,2,36864); SB;
MF(2,1,0,0,33); SB; RW1(6,0,2,37376); MW(117760); SB;
MF(2,0,1,1,32); SB; RW0(7,0,3,36864); DW(3,3,1); SB;
MF(2,1,1,1,33); SB; RW1(7,0,3,37376); SB;
MFW(2,0,2,2,32); SB; RW0(0,0,0,40960); SB;
MF(2,1,2,2,33); SB; RW1(0,0,0,41472); MW(81920); SB;
MF(2,0,3,3,32); SB; RW0(1,0,1,40960); DW(0,3,2); SB;
MF(2,1,3,3,33); SB; RW1(1,0,1,41472); SB;
MFW(2,0,0,4,34); SB; RW0(2,0,2,40960); SB;
MF(2,1,0,4,35); SB; RW1(2,0,2,41472); MW(82944); SB;
MF(2,0,1,5,34); SB; RW0(3,0,3,40960); DW(1,3,2); SB;
MF(2,1,1,5,35); SB; RW1(3,0,3,41472); SB;
MFW(2,0,2,6,34); SB; RW0(4,0,0,45056); SB;
MF(2,1,2,6,35); SB; RW1(4,0,0,45568); MW(83968); SB;
MF(2,0,3,7,34); SB; RW0(5,0,1,45056); DW(2,3,2); SB;
MF(2,1,3,7,35); SB; RW1(5,0,1,45568); SB;
MFW(2,0,0,0,36); SB; RW0(6,0,2,45056); SB;
MF(2,1,0,0,37); SB; RW1(6,0,2,45568); MW(84992); SB;
MF(2,0,1,1,36); SB; RW0(7,0,3,45056); DW(3,3,2); SB;
MF(2,1,1,1,37); SB; RW1(7,0,3,45568); SB;
MFW(2,0,2,2,36); SB; RW0(0,1,0,49152); GW(8); GK0; SB;
MF(2,1,2,2,37); SB; RW1(0,1,0,49664); GK1; GC(1); GM(0,1); SB;
MF(2,0,3,3,36); SB; RW0(1,1,1,49152); GM(1,1); SB;
MF(2,1,3,3,37); SB; RW1(1,1,1,49664); GM(2,1); SB;
MFW(2,0,0,4,38); SB; RW0(2,1,2,49152); GM(3,1); SB;
MF(2,1,0,4,39); SB; RW1(2,1,2,49664); SB;
MF(2,0,1,5,38); SB; RW0(3,1,3,49152); SB;
MF(2,1,1,5,39); SB; RW1(3,1,3,49664); SB;
MFW(2,0,2,6,38); SB; RW0(4,1,0,53248); SB;
MF(2,1,2,6,39); SB; RW1(4,1,0,53760); SB;
MF(2,0,3,7,38); SB; RW0(5,1,1,53248); SB;
MF(2,1,3,7,39); SB; RW1(5,1,1,53760); SB;
MFW(2,0,0,0,40); SB; RW0(6,1,2,53248); GP(0); GP(1); SB;
MF(2,1,0,0,41); SB; RW1(6,1,2,53760); GP(2); GP(3); SB;
MF(2,0,1,1,40); SB; RW0(7,1,3,53248); GT(1); SB;
MF(2,1,1,1,41); SB; RW1(7,1,3,53760); SB;
MFW(2,0,2,2,40); SB; RW0(0,1,0,57344); SB;
MF(2,1,2,2,41); SB; RW1(0,1,0,57856); SB;
MF(2,0,3,3,40); SB; RW0(1,1,1,57344); SB;
MF(2,1,3,3,41); SB; RW1(1,1,1,57856); SB;
MFW(2,0,0,4,42); SB; RW0(2,1,2,57344); SB;
MF(2,1,0,4,43); SB; RW1(2,1,2,57856); SB;
MF(2,0,1,5,42); SB; RW0(3,1,3,57344); SB;
MF(2,1,1,5,43); SB; RW1(3,1,3,57856); SB;
MFW(2,0,2,6,42); SB; RW0(4,1,0,61440); SB;
MF(2,1,2,6,43); SB; RW1(4,1,0,61952); SB;
MF(2,0,3,7,42); SB; RW0(5,1,1,61440); SB;
MF(2,1,3,7,43); SB; RW1(5,1,1,61952); SB;
MFW(2,0,0,0,44); SB; RW0(6,1,2,61440); SB;
MF(2,1,0,0,45); SB; RW1(6,1,2,61952); SB;
MF(2,0,1,1,44); SB; RW0(7,1,3,61440); SB;
MF(2,1,1,1,45); SB; RW1(7,1,3,61952); SB;
WV(16); BAR;
MFW(2,0,2,2,44); SB; RW0(0,0,0,16384); SB;
MF(2,1,2,2,45); SB; RW1(0,0,0,16896); MW(131072); SB;
MF(2,0,3,3,44); SB; RW0(1,0,1,16384); DW(0,3,3); SB;
MF(2,1,3,3,45); SB; RW1(1,0,1,16896); SB;
MFW(2,0,0,4,46); SB; RW0(2,0,2,16384); SB;
MF(2,1,0,4,47); SB; RW1(2,0,2,16896); MW(132096); SB;
MF(2,0,1,5,46); SB; RW0(3,0,3,16384); DW(1,3,3); SB;
MF(2,1,1,5,47); SB; RW1(3,0,3,16896); SB;
MFW(2,0,2,6,46); SB; RW0(4,0,0,20480); SB;
MF(2,1,2,6,47); SB; RW1(4,0,0,20992); MW(133120); SB;
MF(2,0,3,7,46); SB; RW0(5,0,1,20480); DW(2,3,3); SB;
MF(2,1,3,7,47); SB; RW1(5,0,1,20992); SB;
MFW(2,0,0,0,48); SB; RW0(6,0,2,20480); SB;
MF(2,1,0,0,49); SB; RW1(6,0,2,20992); MW(134144); SB;
MF(2,0,1,1,48); SB; RW0(7,0,3,20480); DW(3,3,3); SB;
MF(2,1,1,1,49); SB; RW1(7,0,3,20992); SB;
MFW(2,0,2,2,48); SB; RW0(0,0,0,24576); SB;
MF(2,1,2,2,49); SB; RW1(0,0,0,25088); MW(65536); SB;
MF(2,0,3,3,48); SB; RW0(1,0,1,24576); DW(0,3,4); SB;
MF(2,1,3,3,49); SB; RW1(1,0,1,25088); SB;
MFW(2,0,0,4,50); SB; RW0(2,0,2,24576); SB;
MF(2,1,0,4,51); SB; RW1(2,0,2,25088); MW(66560); SB;
MF(2,0,1,5,50); SB; RW0(3,0,3,24576); DW(1,3,4); SB;
MF(2,1,1,5,51); SB; RW1(3,0,3,25088); SB;
MFW(2,0,2,6,50); SB; RW0(4,0,0,28672); SB;
MF(2,1,2,6,51); SB; RW1(4,0,0,29184); MW(67584); SB;
MF(2,0,3,7,50); SB; RW0(5,0,1,28672); DW(2,3,4); SB;
MF(2,1,3,7,51); SB; RW1(5,0,1,29184); SB;
MFW(2,0,0,0,52); SB; RW0(6,0,2,28672); SB;
MF(2,1,0,0,53); SB; RW1(6,0,2,29184); MW(68608); SB;
MF(2,0,1,1,52); SB; RW0(7,0,3,28672); DW(3,3,4); SB;
MF(2,1,1,1,53); SB; RW1(7,0,3,29184); SB;
MFW(2,0,2,2,52); SB; RW0(0,1,0,0); SB;
MF(2,1,2,2,53); SB; RW1(0,1,0,512); SB;
MF(2,0,3,3,52); SB; RW0(1,1,1,0); SB;
MF(2,1,3,3,53); SB; RW1(1,1,1,512); SB;
MFW(2,0,0,4,54); SB; RW0(2,1,0,4096); SB;
MF(2,1,0,4,55); SB; RW1(2,1,0,4608); SB;
MF(2,0,1,5,54); SB; RW0(3,1,1,4096); SB;
MF(2,1,1,5,55); SB; RW1(3,1,1,4608); SB;
MFW(2,0,2,6,54); SB; RW0(4,1,0,8192); SB;
MF(2,1,2,6,55); SB; RW1(4,1,0,8704); SB;
MF(2,0,3,7,54); SB; RW0(5,1,1,8192); SB;
MF(2,1,3,7,55); SB; RW1(5,1,1,8704); SB;
MFW(2,0,0,0,56); SB; RW0(6,1,0,12288); SB;
MF(2,1,0,0,57); SB; RW1(6,1,0,12800); SB;
MF(2,0,1,1,56); SB; RW0(7,1,1,12288); SB;
MF(2,1,1,1,57); SB; RW1(7,1,1,12800); SB;
MFW(2,0,0,2,58); SB; RW0(0,1,2,0); SB;
MF(2,1,0,2,59); SB; RW1(0,1,2,512); SB;
MF(2,0,1,3,58); SB; RW0(1,1,3,0); SB;
MF(2,1,1,3,59); SB; RW1(1,1,3,512); SB;
MFW(2,0,0,4,60); SB; RW0(2,1,2,4096); SB;
MF(2,1,0,4,61); SB; RW1(2,1,2,4608); SB;
MF(2,0,1,5,60); SB; RW0(3,1,3,4096); SB;
MF(2,1,1,5,61); SB; RW1(3,1,3,4608); SB;
MFW(2,0,0,6,62); SB; RW0(4,1,2,8192); SB;
MF(2,1,0,6,63); SB; RW1(4,1,2,8704); SB;
MF(2,0,1,7,62); SB; RW0(5,1,3,8192); SB;
MF(2,1,1,7,63); SB; RW1(5,1,3,8704); SB;
MFW(2,0,2,0,56); SB; RW0(6,1,2,12288); SB;
MF(2,1,2,0,57); SB; RW1(6,1,2,12800); SB;
MF(2,0,3,1,56); SB; RW0(7,1,3,12288); SB;
MF(2,1,3,1,57); SB; RW1(7,1,3,12800); SB;
WV(13); BAR;
MFW(2,0,2,2,58); SB; RW0(0,0,0,0); PK0(2,0,0); PK1(2,0,0); SB;
MF(2,1,2,2,59); SB; RW1(0,0,0,512); PK2(2,0,0); PK3(2,0,0); SB;
MF(2,0,3,3,58); SB; RW0(1,0,1,0); PK4(2,0,0); PK5(2,0,0); SB;
MF(2,1,3,3,59); SB; RW1(1,0,1,512); PK6(2,0,0); SB;
MFW(2,0,2,4,60); SB; RW0(2,0,2,0); PK0(2,1,0); PK1(2,1,0); SB;
MF(2,1,2,4,61); SB; RW1(2,0,2,512); PK2(2,1,0); PK3(2,1,0); SB;
MF(2,0,3,5,60); SB; RW0(3,0,3,0); PK4(2,1,0); PK5(2,1,0); SB;
MF(2,1,3,5,61); SB; RW1(3,0,3,512); PK6(2,1,0); SB;
MFW(2,0,2,6,62); SB; RW0(4,0,0,4096); SB;
MF(2,1,2,6,63); SB; RW1(4,0,0,4608); MW(147456); SB;
MF(2,0,3,7,62); SB; RW0(5,0,1,4096); DW(0,3,5); SB;
MF(2,1,3,7,63); SB; RW1(5,0,1,4608); SB;
MZ(3,0,0,0,0); SB; RW0(6,0,2,4096); SB;
MZ(3,1,0,0,1); SB; RW1(6,0,2,4608); MW(148480); SB;
MZ(3,0,1,1,0); SB; RW0(7,0,3,4096); DW(1,3,5); SB;
MZ(3,1,1,1,1); SB; RW1(7,0,3,4608); SB;
MZ(3,0,2,2,0); SB; RW0(0,0,0,8192); PK0(2,0,1); PK1(2,0,1); SB;
MZ(3,1,2,2,1); SB; RW1(0,0,0,8704); PK2(2,0,1); PK3(2,0,1); SB;
MZ(3,0,3,3,0); SB; RW0(1,0,1,8192); PK4(2,0,1); PK5(2,0,1); SB;
MZ(3,1,3,3,1); SB; RW1(1,0,1,8704); PK6(2,0,1); SB;
MFW(3,0,0,4,2); SB; RW0(2,0,2,8192); PK0(2,1,1); PK1(2,1,1); SB;
MF(3,1,0,4,3); SB; RW1(2,0,2,8704); PK2(2,1,1); PK3(2,1,1); SB;
MF(3,0,1,5,2); SB; RW0(3,0,3,8192); PK4(2,1,1); PK5(2,1,1); SB;
MF(3,1,1,5,3); SB; RW1(3,0,3,8704); PK6(2,1,1); SB;
MFW(3,0,2,6,2); SB; RW0(4,0,0,12288); SB;
MF(3,1,2,6,3); SB; RW1(4,0,0,12800); MW(149504); SB;
MF(3,0,3,7,2); SB; RW0(5,0,1,12288); DW(2,3,5); SB;
MF(3,1,3,7,3); SB; RW1(5,0,1,12800); SB;
MFW(3,0,0,0,4); SB; RW0(6,0,2,12288); SB;
MF(3,1,0,0,5); SB; RW1(6,0,2,12800); MW(150528); SB;
MF(3,0,1,1,4); SB; RW0(7,0,3,12288); DW(3,3,5); SB;
MF(3,1,1,1,5); SB; RW1(7,0,3,12800); SB;
MFW(3,0,2,2,4); SB; RW0(0,1,0,16384); SB;
MF(3,1,2,2,5); SB; RW1(0,1,0,16896); MW(49152); SB;
MF(3,0,3,3,4); SB; RW0(1,1,1,16384); DW(0,3,6); SB;
MF(3,1,3,3,5); SB; RW1(1,1,1,16896); SB;
MFW(3,0,0,4,6); SB; RW0(2,1,2,16384); SB;
MF(3,1,0,4,7); SB; RW1(2,1,2,16896); MW(50176); SB;
MF(3,0,1,5,6); SB; RW0(3,1,3,16384); DW(1,3,6); SB;
MF(3,1,1,5,7); SB; RW1(3,1,3,16896); SB;
BAR;
MFW(3,0,2,6,6); SB; RW0(4,1,0,20480); GO(2); SB;
MF(3,1,2,6,7); SB; RW1(4,1,0,20992); MW(51200); SB;
MF(3,0,3,7,6); SB; RW0(5,1,1,20480); DW(2,3,6); SB;
MF(3,1,3,7,7); SB; RW1(5,1,1,20992); SB;
MFW(3,0,0,0,8); SB; RW0(6,1,2,20480); SB;
MF(3,1,0,0,9); SB; RW1(6,1,2,20992); MW(52224); SB;
MF(3,0,1,1,8); SB; RW0(7,1,3,20480); DW(3,3,6); SB;
MF(3,1,1,1,9); SB; RW1(7,1,3,20992); SB;
MFW(3,0,2,2,8); SB; RW0(0,1,0,24576); SB;
MF(3,1,2,2,9); SB; RW1(0,1,0,25088); MW(98304); SB;
MF(3,0,3,3,8); SB; RW0(1,1,1,24576); DW(0,3,7); SB;
MF(3,1,3,3,9); SB; RW1(1,1,1,25088); SB;
MFW(3,0,0,4,10); SB; RW0(2,1,2,24576); SB;
MF(3,1,0,4,11); SB; RW1(2,1,2,25088); MW(99328); SB;
MF(3,0,1,5,10); SB; RW0(3,1,3,24576); DW(1,3,7); SB;
MF(3,1,1,5,11); SB; RW1(3,1,3,25088); SB;
MFW(3,0,2,6,10); SB; RW0(4,1,0,28672); SB;
MF(3,1,2,6,11); SB; RW1(4,1,0,29184); MW(100352); SB;
MF(3,0,3,7,10); SB; RW0(5,1,1,28672); DW(2,3,7); SB;
MF(3,1,3,7,11); SB; RW1(5,1,1,29184); SB;
MFW(3,0,0,0,12); SB; RW0(6,1,2,28672); SB;
MF(3,1,0,0,13); SB; RW1(6,1,2,29184); MW(101376); SB;
MF(3,0,1,1,12); SB; RW0(7,1,3,28672); DW(3,3,7); SB;
MF(3,1,1,1,13); SB; RW1(7,1,3,29184); SB;
WV(16); BAR;
MFW(3,0,2,2,12); SB; RW0(0,0,0,49152); SB;
MF(3,1,2,2,13); SB; RW1(0,0,0,49664); SB;
MF(3,0,3,3,12); SB; RW0(1,0,1,49152); SB;
MF(3,1,3,3,13); SB; RW1(1,0,1,49664); SB;
MFW(3,0,0,4,14); SB; RW0(2,0,2,49152); SB;
MF(3,1,0,4,15); SB; RW1(2,0,2,49664); SB;
MF(3,0,1,5,14); SB; RW0(3,0,3,49152); SB;
MF(3,1,1,5,15); SB; RW1(3,0,3,49664); SB;
MFW(3,0,2,6,14); SB; RW0(4,0,0,53248); GL(2,0); SB;
MF(3,1,2,6,15); SB; RW1(4,0,0,53760); GL(2,1); SB;
MF(3,0,3,7,14); SB; RW0(5,0,1,53248); GL(2,2); SB;
MF(3,1,3,7,15); SB; RW1(5,0,1,53760); SB;
MFW(3,0,0,0,16); SB; RW0(6,0,2,53248); SB;
MF(3,1,0,0,17); SB; RW1(6,0,2,53760); SB;
MF(3,0,1,1,16); SB; RW0(7,0,3,53248); SB;
MF(3,1,1,1,17); SB; RW1(7,0,3,53760); SB;
MFW(3,0,2,2,16); SB; RW0(0,0,0,57344); SB;
MF(3,1,2,2,17); SB; RW1(0,0,0,57856); SB;
MF(3,0,3,3,16); SB; RW0(1,0,1,57344); SB;
MF(3,1,3,3,17); SB; RW1(1,0,1,57856); SB;
MFW(3,0,0,4,18); SB; RW0(2,0,2,57344); SB;
MF(3,1,0,4,19); SB; RW1(2,0,2,57856); SB;
MF(3,0,1,5,18); SB; RW0(3,0,3,57344); SB;
MF(3,1,1,5,19); SB; RW1(3,0,3,57856); SB;
MFW(3,0,2,6,18); SB; RW0(4,0,0,61440); SB;
MF(3,1,2,6,19); SB; RW1(4,0,0,61952); SB;
MF(3,0,3,7,18); SB; RW0(5,0,1,61440); SB;
MF(3,1,3,7,19); SB; RW1(5,0,1,61952); SB;
MFW(3,0,0,0,20); SB; RW0(6,0,2,61440); SB;
MF(3,1,0,0,21); SB; RW1(6,0,2,61952); SB;
MF(3,0,1,1,20); SB; RW0(7,0,3,61440); SB;
MF(3,1,1,1,21); SB; RW1(7,0,3,61952); SB;
MFW(3,0,2,2,20); SB; RW0(0,1,0,32768); SB;
MF(3,1,2,2,21); SB; RW1(0,1,0,33280); SB;
MF(3,0,3,3,20); SB; RW0(1,1,1,32768); SB;
MF(3,1,3,3,21); SB; RW1(1,1,1,33280); SB;
MFW(3,0,0,4,22); SB; RW0(2,1,2,32768); SB;
MF(3,1,0,4,23); SB; RW1(2,1,2,33280); SB;
MF(3,0,1,5,22); SB; RW0(3,1,3,32768); SB;
MF(3,1,1,5,23); SB; RW1(3,1,3,33280); SB;
MFW(3,0,2,6,22); SB; RW0(4,1,0,36864); SB;
MF(3,1,2,6,23); SB; RW1(4,1,0,37376); SB;
MF(3,0,3,7,22); SB; RW0(5,1,1,36864); SB;
MF(3,1,3,7,23); SB; RW1(5,1,1,37376); SB;
MFW(3,0,0,0,24); SB; RW0(6,1,2,36864); SB;
MF(3,1,0,0,25); SB; RW1(6,1,2,37376); SB;
MF(3,0,1,1,24); SB; RW0(7,1,3,36864); SB;
MF(3,1,1,1,25); SB; RW1(7,1,3,37376); SB;
MFW(3,0,2,2,24); SB; RW0(0,1,0,40960); GW(0); GK0; SB;
MF(3,1,2,2,25); SB; RW1(0,1,0,41472); GK1; GC(2); GM(0,0); SB;
MF(3,0,3,3,24); SB; RW0(1,1,1,40960); GM(1,0); SB;
MF(3,1,3,3,25); SB; RW1(1,1,1,41472); GM(2,0); SB;
MFW(3,0,0,4,26); SB; RW0(2,1,2,40960); GM(3,0); SB;
MF(3,1,0,4,27); SB; RW1(2,1,2,41472); SB;
MF(3,0,1,5,26); SB; RW0(3,1,3,40960); SB;
MF(3,1,1,5,27); SB; RW1(3,1,3,41472); SB;
MFW(3,0,2,6,26); SB; RW0(4,1,0,45056); SB;
MF(3,1,2,6,27); SB; RW1(4,1,0,45568); SB;
MF(3,0,3,7,26); SB; RW0(5,1,1,45056); SB;
MF(3,1,3,7,27); SB; RW1(5,1,1,45568); SB;
MFW(3,0,0,0,28); SB; RW0(6,1,2,45056); GP(0); GP(1); SB;
MF(3,1,0,0,29); SB; RW1(6,1,2,45568); GP(2); GP(3); SB;
MF(3,0,1,1,28); SB; RW0(7,1,3,45056); GT(2); SB;
MF(3,1,1,1,29); SB; RW1(7,1,3,45568); SB;
WV(12); BAR;
MFW(3,0,2,2,28); SB; RW0(0,0,0,32768); SB;
MF(3,1,2,2,29); SB; RW1(0,0,0,33280); SB;
MF(3,0,3,3,28); SB; RW0(1,0,1,32768); SB;
MF(3,1,3,3,29); SB; RW1(1,0,1,33280); SB;
MFW(3,0,0,4,30); SB; RW0(2,0,2,32768); SB;
MF(3,1,0,4,31); SB; RW1(2,0,2,33280); SB;
MF(3,0,1,5,30); SB; RW0(3,0,3,32768); SB;
MF(3,1,1,5,31); SB; RW1(3,0,3,33280); SB;
MFW(3,0,2,6,30); SB; RW0(4,0,0,36864); SB;
MF(3,1,2,6,31); SB; RW1(4,0,0,37376); SB;
MF(3,0,3,7,30); SB; RW0(5,0,1,36864); SB;
MF(3,1,3,7,31); SB; RW1(5,0,1,37376); SB;
MFW(3,0,0,0,32); SB; RW0(6,0,2,36864); SB;
MF(3,1,0,0,33); SB; RW1(6,0,2,37376); SB;
MF(3,0,1,1,32); SB; RW0(7,0,3,36864); SB;
MF(3,1,1,1,33); SB; RW1(7,0,3,37376); SB;
MFW(3,0,2,2,32); SB; RW0(0,0,0,40960); SB;
MF(3,1,2,2,33); SB; RW1(0,0,0,41472); SB;
MF(3,0,3,3,32); SB; RW0(1,0,1,40960); SB;
MF(3,1,3,3,33); SB; RW1(1,0,1,41472); SB;
MFW(3,0,0,4,34); SB; RW0(2,0,2,40960); SB;
MF(3,1,0,4,35); SB; RW1(2,0,2,41472); SB;
MF(3,0,1,5,34); SB; RW0(3,0,3,40960); SB;
MF(3,1,1,5,35); SB; RW1(3,0,3,41472); SB;
MFW(3,0,2,6,34); SB; RW0(4,0,0,45056); SB;
MF(3,1,2,6,35); SB; RW1(4,0,0,45568); SB;
MF(3,0,3,7,34); SB; RW0(5,0,1,45056); SB;
MF(3,1,3,7,35); SB; RW1(5,0,1,45568); SB;
MFW(3,0,0,0,36); SB; RW0(6,0,2,45056); SB;
MF(3,1,0,0,37); SB; RW1(6,0,2,45568); SB;
MF(3,0,1,1,36); SB; RW0(7,0,3,45056); SB;
MF(3,1,1,1,37); SB; RW1(7,0,3,45568); SB;
MFW(3,0,2,2,36); SB; RW0(0,1,0,49152); SB;
MF(3,1,2,2,37); SB; RW1(0,1,0,49664); SB;
MF(3,0,3,3,36); SB; RW0(1,1,1,49152); SB;
MF(3,1,3,3,37); SB; RW1(1,1,1,49664); SB;
MFW(3,0,0,4,38); SB; RW0(2,1,2,49152); SB;
MF(3,1,0,4,39); SB; RW1(2,1,2,49664); SB;
MF(3,0,1,5,38); SB; RW0(3,1,3,49152); SB;
MF(3,1,1,5,39); SB; RW1(3,1,3,49664); SB;
MFW(3,0,2,6,38); SB; RW0(4,1,0,53248); SB;
MF(3,1,2,6,39); SB; RW1(4,1,0,53760); SB;
MF(3,0,3,7,38); SB; RW0(5,1,1,53248); SB;
MF(3,1,3,7,39); SB; RW1(5,1,1,53760); SB;
MFW(3,0,0,0,40); SB; RW0(6,1,2,53248); SB;
MF(3,1,0,0,41); SB; RW1(6,1,2,53760); SB;
MF(3,0,1,1,40); SB; RW0(7,1,3,53248); SB;
MF(3,1,1,1,41); SB; RW1(7,1,3,53760); SB;
MFW(3,0,2,2,40); SB; RW0(0,1,0,57344); SB;
MF(3,1,2,2,41); SB; RW1(0,1,0,57856); SB;
MF(3,0,3,3,40); SB; RW0(1,1,1,57344); SB;
MF(3,1,3,3,41); SB; RW1(1,1,1,57856); SB;
MFW(3,0,0,4,42); SB; RW0(2,1,2,57344); SB;
MF(3,1,0,4,43); SB; RW1(2,1,2,57856); SB;
MF(3,0,1,5,42); SB; RW0(3,1,3,57344); SB;
MF(3,1,1,5,43); SB; RW1(3,1,3,57856); SB;
MFW(3,0,2,6,42); SB; RW0(4,1,0,61440); SB;
MF(3,1,2,6,43); SB; RW1(4,1,0,61952); SB;
MF(3,0,3,7,42); SB; RW0(5,1,1,61440); SB;
MF(3,1,3,7,43); SB; RW1(5,1,1,61952); SB;
MFW(3,0,0,0,44); SB; RW0(6,1,2,61440); SB;
MF(3,1,0,0,45); SB; RW1(6,1,2,61952); SB;
MF(3,0,1,1,44); SB; RW0(7,1,3,61440); SB;
MF(3,1,1,1,45); SB; RW1(7,1,3,61952); SB;
WV(4); BAR;
MFW(3,0,2,2,44); SB; RW0(0,0,0,16384); SB;
MF(3,1,2,2,45); SB; RW1(0,0,0,16896); SB;
MF(3,0,3,3,44); SB; RW0(1,0,1,16384); SB;
MF(3,1,3,3,45); SB; RW1(1,0,1,16896); SB;
MFW(3,0,0,4,46); SB; RW0(2,0,2,16384); SB;
MF(3,1,0,4,47); SB; RW1(2,0,2,16896); SB;
MF(3,0,1,5,46); SB; RW0(3,0,3,16384); SB;
MF(3,1,1,5,47); SB; RW1(3,0,3,16896); SB;
MFW(3,0,2,6,46); SB; RW0(4,0,0,20480); SB;
MF(3,1,2,6,47); SB; RW1(4,0,0,20992); SB;
MF(3,0,3,7,46); SB; RW0(5,0,1,20480); SB;
MF(3,1,3,7,47); SB; RW1(5,0,1,20992); SB;
MFW(3,0,0,0,48); SB; RW0(6,0,2,20480); SB;
MF(3,1,0,0,49); SB; RW1(6,0,2,20992); SB;
MF(3,0,1,1,48); SB; RW0(7,0,3,20480); SB;
MF(3,1,1,1,49); SB; RW1(7,0,3,20992); SB;
MFW(3,0,2,2,48); SB; RW0(0,0,0,24576); SB;
MF(3,1,2,2,49); SB; RW1(0,0,0,25088); SB;
MF(3,0,3,3,48); SB; RW0(1,0,1,24576); SB;
MF(3,1,3,3,49); SB; RW1(1,0,1,25088); SB;
MFW(3,0,0,4,50); SB; RW0(2,0,2,24576); SB;
MF(3,1,0,4,51); SB; RW1(2,0,2,25088); SB;
MF(3,0,1,5,50); SB; RW0(3,0,3,24576); SB;
MF(3,1,1,5,51); SB; RW1(3,0,3,25088); SB;
MFW(3,0,2,6,50); SB; RW0(4,0,0,28672); SB;
MF(3,1,2,6,51); SB; RW1(4,0,0,29184); SB;
MF(3,0,3,7,50); SB; RW0(5,0,1,28672); SB;
MF(3,1,3,7,51); SB; RW1(5,0,1,29184); SB;
MFW(3,0,0,0,52); SB; RW0(6,0,2,28672); SB;
MF(3,1,0,0,53); SB; RW1(6,0,2,29184); SB;
MF(3,0,1,1,52); SB; RW0(7,0,3,28672); SB;
MF(3,1,1,1,53); SB; RW1(7,0,3,29184); SB;
MFW(3,0,2,2,52); SB; RW0(0,1,0,0); SB;
MF(3,1,2,2,53); SB; RW1(0,1,0,512); SB;
MF(3,0,3,3,52); SB; RW0(1,1,1,0); SB;
MF(3,1,3,3,53); SB; RW1(1,1,1,512); SB;
MFW(3,0,0,4,54); SB; RW0(2,1,0,4096); SB;
MF(3,1,0,4,55); SB; RW1(2,1,0,4608); SB;
MF(3,0,1,5,54); SB; RW0(3,1,1,4096); SB;
MF(3,1,1,5,55); SB; RW1(3,1,1,4608); SB;
MFW(3,0,2,6,54); SB; RW0(4,1,0,8192); SB;
MF(3,1,2,6,55); SB; RW1(4,1,0,8704); SB;
MF(3,0,3,7,54); SB; RW0(5,1,1,8192); SB;
MF(3,1,3,7,55); SB; RW1(5,1,1,8704); SB;
MFW(3,0,0,0,56); SB; RW0(6,1,0,12288); SB;
MF(3,1,0,0,57); SB; RW1(6,1,0,12800); SB;
MF(3,0,1,1,56); SB; RW0(7,1,1,12288); SB;
MF(3,1,1,1,57); SB; RW1(7,1,1,12800); SB;
MFW(3,0,0,2,58); SB; RW0(0,1,2,0); SB;
MF(3,1,0,2,59); SB; RW1(0,1,2,512); SB;
MF(3,0,1,3,58); SB; RW0(1,1,3,0); SB;
MF(3,1,1,3,59); SB; RW1(1,1,3,512); SB;
MFW(3,0,0,4,60); SB; RW0(2,1,2,4096); SB;
MF(3,1,0,4,61); SB; RW1(2,1,2,4608); SB;
MF(3,0,1,5,60); SB; RW0(3,1,3,4096); SB;
MF(3,1,1,5,61); SB; RW1(3,1,3,4608); SB;
MFW(3,0,0,6,62); SB; RW0(4,1,2,8192); SB;
MF(3,1,0,6,63); SB; RW1(4,1,2,8704); SB;
MF(3,0,1,7,62); SB; RW0(5,1,3,8192); SB;
MF(3,1,1,7,63); SB; RW1(5,1,3,8704); SB;
MFW(3,0,2,0,56); SB; RW0(6,1,2,12288); SB;
MF(3,1,2,0,57); SB; RW1(6,1,2,12800); SB;
MF(3,0,3,1,56); SB; RW0(7,1,3,12288); SB;
MF(3,1,3,1,57); SB; RW1(7,1,3,12800); SB;
MFW(3,0,2,2,58); SB;  PK0(3,0,0); PK1(3,0,0); PK2(3,0,0); SB;
MF(3,1,2,2,59); SB;  PK3(3,0,0); PK4(3,0,0); PK5(3,0,0); SB;
MF(3,0,3,3,58); SB;  PK6(3,0,0); PK0(3,1,0); SB;
MF(3,1,3,3,59); SB;  PK1(3,1,0); PK2(3,1,0); SB;
MFW(3,0,2,4,60); SB;  PK3(3,1,0); PK4(3,1,0); PK5(3,1,0); SB;
MF(3,1,2,4,61); SB;  PK6(3,1,0); SB;
MF(3,0,3,5,60); SB;
MF(3,1,3,5,61); SB;
MFW(3,0,2,6,62); SB;
MF(3,1,2,6,63); SB;
MF(3,0,3,7,62); SB;
MF(3,1,3,7,63); SB;
NOPACC;
PK0(3,0,1);
PK1(3,0,1);
PK2(3,0,1);
PK3(3,0,1);
PK4(3,0,1);
PK5(3,0,1);
PK6(3,0,1);
PK0(3,1,1);
PK1(3,1,1);
PK2(3,1,1);
PK3(3,1,1);
PK4(3,1,1);
PK5(3,1,1);
PK6(3,1,1);
WL; BAR; GO(3);
TDELAY;
GL(3,0);
GL(3,1);
GL(3,2);
GW(0);
GK0;
GK1;
GC(3);
GM(0,1);
GM(1,1);
GM(2,1);
GM(3,1);
NOP8;
GP(0);
GP(1);
GP(2);
GP(3);
GT(3);

  if (dstrel != 7) {
    WV(0);
    if (lane == 0) __hip_atomic_store(flp, call, __ATOMIC_RELAXED, __HIP_MEMORY_SCOPE_SYSTEM);
  } else if (wid == 0) {
    const bool mine = ((lane & 31) >> 2) != rank;
    const unsigned long long t0 = __builtin_amdgcn_s_memrealtime();
    for (unsigned n = 1;; ++n) {
      const unsigned v = __hip_atomic_load(flp, __ATOMIC_RELAXED, __HIP_MEMORY_SCOPE_SYSTEM);
      if (__builtin_amdgcn_ballot_w64(mine && (int)(v - call) < 0) == 0) break;
      if ((n & 15u) == 0 && GIVE_UP(t0)) {
        __hip_atomic_store(tmo, call, __ATOMIC_RELAXED, __HIP_MEMORY_SCOPE_AGENT);
        if (lane == 0) *(GAS unsigned short*)((GAS char*)outp + out_v + COFF(0)) = 0x7fc0;
        break;
      }
      __builtin_amdgcn_s_sleep(1);
    }
  }
  if (tid == 0) ((GAS unsigned*)wsl)[WS_EPOCH / 4 + bid] = call;
}
}

extern "C" void kernel_launch(int rank, int nranks, ncclComm_t comm, hipStream_t stream,
                              void* const* d_in, const int* in_sizes, int n_in,
                              void* d_out, int out_size, const int* in_dtypes, int out_dtype,
                              void* const* d_out_all, void* const* d_ws_all, size_t ws_bytes) {
  if (nranks != NR || (unsigned)rank >= (unsigned)NR || ws_bytes < WS_NEED || n_in < 2 || in_sizes[0] != MS * KD ||
      in_sizes[1] != KD * ND || out_size != NR * MS * NS) { fprintf(stderr, "unsupported configuration\n"); return; }
  Params p;
  p.x = d_in[0]; p.w = d_in[1]; p.rank = rank; p.wsl = (unsigned*)d_ws_all[rank];
  for (int i = 0; i < NR; ++i) { p.out[i] = d_out_all[i]; p.ws[i] = d_ws_all[i]; }
  gemm_a2a<<<NWG, 256, 0, stream>>>(p);
  HIP_CHECK(hipGetLastError());
}
```
